```python
import numpy as np

kernel_name = "gpu_gemm_ar_m4096_k4096_n4096_f32_relu"


ABSMAX_RTOL = 0.001

COMPUTE_DTYPE = "float32"
EPILOGUE = "relu"
IN_QUANT = None
OUT_QUANT = None
IN_DTYPES = {"x": "float32", "w_mat": "float32"}
WORLD_SIZE = 8

def setup_inputs():

    x = np.random.randn(4096, 4096).astype(np.float32)
    w_mat = (np.random.randn(4096, 4096) / np.sqrt(4096)).astype(np.float32)
    return {"x": x, "w_mat": w_mat}

def _epilogue(y):
    return np.maximum(y, 0.0).astype(np.float32)

def reference(x, w_mat):
    y = (x @ w_mat).astype(np.float32)
    return _epilogue(y)
```

```cpp
#include <hip/hip_runtime.h>
#include <rccl/rccl.h>
#include <cstdio>
#include <cstdint>

#define HIP_CHECK(expr) do { hipError_t _e = (expr); if (_e != hipSuccess) { \
    fprintf(stderr, "HIP error %s at %s:%d\n", hipGetErrorString(_e), __FILE__, __LINE__); \
    return; } } while (0)

typedef unsigned u32x4 __attribute__((ext_vector_type(4)));
typedef float f32x4 __attribute__((ext_vector_type(4)));
typedef float f32x16 __attribute__((ext_vector_type(16)));
typedef _Float16 f16x8 __attribute__((ext_vector_type(8)));
typedef unsigned long long u64;
typedef __attribute__((address_space(1))) unsigned gu32;

constexpr int NR = 8;
constexpr int MM = 4096, NN = 4096, KS = 512;
constexpr int NWG = 256, NTHR = 512;
#define NSEND 128
static_assert(NSEND >= 16 && NSEND <= 256, "one LDS counter per round");
constexpr int NSLOT = 4;
constexpr int PD = NSLOT - 1;
constexpr int REG_B = 8704;
constexpr int LDS_BYTES = 132 * 1024;
constexpr int LDS_CNT = 131 * 1024;
constexpr u64 DEADLINE = 40000000ull;

constexpr size_t OFF_SEQ = 0;
constexpr size_t OFF_TMO = 32768;
constexpr size_t OFF_XF = 65536;
constexpr size_t OFF_WF = OFF_XF + 8 * 256;
constexpr size_t OFF_DBG = 262144;
constexpr size_t OFF_X = 1u << 20;
constexpr size_t OFF_W = OFF_X + (8u << 20);
constexpr size_t OFF_O = OFF_W + (16u << 20);
constexpr size_t SLOT_B = 8704;
constexpr size_t WS_NEED = OFF_O + (size_t)NR * 1024 * SLOT_B;

struct Params {
    const float* x;
    const float* w;
    float* out;
    char* ws[NR];
    int rank;
};

__device__ __forceinline__ void st16_sys(void* p, u32x4 v) {
    asm volatile("global_store_dwordx4 %0, %1, off sc0 sc1\n\ts_nop 1" :: "v"(p), "v"(v) : "memory");
}
__device__ __forceinline__ void st16_out(void* p, f32x4 v) {
    asm volatile("global_store_dwordx4 %0, %1, off sc1\n\ts_nop 1" :: "v"(p), "v"(v) : "memory");
}
__device__ __forceinline__ void vm_wait0() { asm volatile("s_waitcnt vmcnt(0)" ::: "memory"); }
__device__ __forceinline__ unsigned ld_sys(const void* p) {
    return __hip_atomic_load((gu32*)p, __ATOMIC_RELAXED, __HIP_MEMORY_SCOPE_SYSTEM);
}
__device__ __forceinline__ void st_sys(void* p, unsigned v) {
    __hip_atomic_store((gu32*)p, v, __ATOMIC_RELAXED, __HIP_MEMORY_SCOPE_SYSTEM);
}
__device__ __forceinline__ unsigned ld_agt(const void* p) {
    return __hip_atomic_load((gu32*)p, __ATOMIC_RELAXED, __HIP_MEMORY_SCOPE_AGENT);
}
__device__ __forceinline__ void st_agt(void* p, unsigned v) {
    __hip_atomic_store((gu32*)p, v, __ATOMIC_RELAXED, __HIP_MEMORY_SCOPE_AGENT);
}
__device__ __forceinline__ void glds4k(const char* sbase, unsigned voff, unsigned lds_dst) {
    unsigned keep;
    asm volatile("s_mov_b32 %0, m0\n\ts_mov_b32 m0, %3\n\ts_nop 3\n\t"
                 "global_load_lds_dwordx4 %1, %2 sc0 sc1\n\t"
                 "global_load_lds_dwordx4 %1, %2 offset:1024 sc0 sc1\n\t"
                 "global_load_lds_dwordx4 %1, %2 offset:2048 sc0 sc1\n\t"
                 "global_load_lds_dwordx4 %1, %2 offset:3072 sc0 sc1\n\t"
                 "s_mov_b32 m0, %0"
                 : "=&s"(keep) : "v"(voff), "s"(sbase), "s"(lds_dst) : "memory");
}
__device__ __forceinline__ void glds1k(const char* sbase, unsigned voff, unsigned lds_dst) {
    unsigned keep;
    asm volatile("s_mov_b32 %0, m0\n\ts_mov_b32 m0, %3\n\ts_nop 3\n\t"
                 "global_load_lds_dwordx4 %1, %2 sc0 sc1\n\t"
                 "s_mov_b32 m0, %0"
                 : "=&s"(keep) : "v"(voff), "s"(sbase), "s"(lds_dst) : "memory");
}
__device__ __forceinline__ unsigned mbcnt64(u64 m) {
    return __builtin_amdgcn_mbcnt_hi((unsigned)(m >> 32), __builtin_amdgcn_mbcnt_lo((unsigned)m, 0u));
}
__device__ __forceinline__ const char* uni_ptr(const char* p) {
    u64 v = (u64)p;
    unsigned lo = __builtin_amdgcn_readfirstlane((unsigned)v), hi = __builtin_amdgcn_readfirstlane((unsigned)(v >> 32));
    return (const char*)(((u64)hi << 32) | lo);
}
__device__ __forceinline__ u32x4 cvt8(f32x4 a, f32x4 b) {
    f16x8 h;
    h[0] = (_Float16)a[0]; h[1] = (_Float16)a[1]; h[2] = (_Float16)a[2]; h[3] = (_Float16)a[3];
    h[4] = (_Float16)b[0]; h[5] = (_Float16)b[1]; h[6] = (_Float16)b[2]; h[7] = (_Float16)b[3];
    return __builtin_bit_cast(u32x4, h);
}

#define STAMP(i) do {} while (0)

struct Blk { int isw, grp, lb, t; };
__device__ __forceinline__ Blk decode_blk(int v) {
    Blk b;
    if (v < 384) {
        b.t = v / 48; int e = v % 48, e3 = e / 3, r3 = e % 3;
        if (r3 == 2) { b.isw = 1; b.grp = e3 & 1; b.lb = e3 >> 1; }
        else { int idx = e3 * 2 + r3; b.isw = 0; b.grp = idx & 3; b.lb = idx >> 2; }
    } else {
        int u = v - 384; b.t = u >> 4; int e = u & 15;
        b.isw = 1; b.grp = e & 1; b.lb = 8 + (e >> 1);
    }
    return b;
}

__global__ void __launch_bounds__(NTHR, 2) gemm_ar_kernel(Params p) {
    extern __shared__ __attribute__((aligned(16))) char lds[];
    const int tid = threadIdx.x, lane = tid & 63;
    const int wv = __builtin_amdgcn_readfirstlane(tid >> 6);
    const int rank = p.rank;
    char* const myws = p.ws[rank];
    const int xcd = blockIdx.x & 7, u = blockIdx.x >> 3;
    const unsigned lds0 = (unsigned)(uintptr_t)(__attribute__((address_space(3))) char*)lds;
    char* const tmo = myws + OFF_TMO;
    STAMP(0);

    const unsigned call = __builtin_amdgcn_readfirstlane(ld_agt(myws + OFF_SEQ + blockIdx.x * 128)) + 1;
    if (tid < 64) *(unsigned*)(lds + LDS_CNT + tid * 4) = 0;
    __syncthreads();
    if (tid == 0) st_agt(myws + OFF_SEQ + blockIdx.x * 128, call);

    if (u < 16) {
        const int stage = u >> 3, ul = u & 7;
        const int rb = (xcd >> 1) * 2 + (ul >> 2);
        const int cb = stage * 8 + (xcd & 1) * 4 + (ul & 3);
        const int wm = wv & 3, kh = wv >> 2;
        const char* const gsrc = (wv < 4) ? myws + OFF_X + (size_t)rb * (64 * 16384)
                                          : myws + OFF_W + (size_t)cb * (64 * 16384);
        const unsigned voff = (wv & 3) * 4096 + lane * 16;
        const int fpos = (lane & ~7) | ((rank + lane) & 7);
        const char* const xf = myws + OFF_XF + rb * 256 + fpos * 4;
        const char* const wf = myws + OFF_WF + cb * 256 + fpos * 4;
        u64 ready = 0;
        auto ensure = [&](int n) {
            if ((ready >> n) & 1) return;
            const u64 t0 = __builtin_amdgcn_s_memrealtime();
            while (true) {
                unsigned a = ld_sys(xf), b = ld_sys(wf);
                ready = __builtin_amdgcn_ballot_w64(a >= call && b >= call);
                if ((ready >> n) & 1) break;
                if (__builtin_amdgcn_readfirstlane(ld_agt(tmo)) == call ||
                    __builtin_amdgcn_s_memrealtime() - t0 > DEADLINE) {
                    if (lane == 0) st_agt(tmo, call);
                    vm_wait0();
                    ready = ~0ull;
                    break;
                }
                __builtin_amdgcn_s_sleep(8);
            }
        };
        auto issue = [&](int n) {
            const int kb = (n & ~7) | ((rank + n) & 7);
            glds4k(gsrc + (size_t)kb * 16384, voff, lds0 + (n % NSLOT) * 32768 + wv * 4096);
        };
        f32x16 acc[4];
#pragma unroll
        for (int c = 0; c < 4; ++c)
#pragma unroll
            for (int r = 0; r < 16; ++r) acc[c][r] = 0.f;

        if (wv == 0) { for (int n = 0; n < PD; ++n) ensure(n); }
        __builtin_amdgcn_s_barrier();
        asm volatile("" ::: "memory");
        STAMP(1);
        for (int n = 0; n < PD; ++n) issue(n);
        for (int n = 0; n < 64; ++n) {
            if (wv == 0 && n + PD < 64) ensure(n + PD);
            if (n + PD <= 64) asm volatile("s_waitcnt vmcnt(%0)" :: "i"(4 * (PD - 1)) : "memory");
            else vm_wait0();
            __builtin_amdgcn_s_barrier();
            asm volatile("" ::: "memory");
            if (n + PD < 64) issue(n + PD);
            const char* sa = lds + (n % NSLOT) * 32768 + lane * 16;
#pragma unroll
            for (int qq = 0; qq < 2; ++qq) {
                const int q = kh * 2 + qq;
                const f16x8 a = *(const f16x8*)(sa + (wm * 4 + q) * 1024);
#pragma unroll
                for (int c = 0; c < 4; ++c) {
                    const f16x8 b = *(const f16x8*)(sa + 16384 + (q * 4 + c) * 1024);
                    acc[c] = __builtin_amdgcn_mfma_f32_32x32x16_f16(a, b, acc[c], 0, 0, 0);
                }
            }
        }
        STAMP(2);
        __syncthreads();
        f32x4 v[8];
        {
            char* dst = lds + (wv ^ 4) * REG_B + lane * 16;
            if (kh == 0) {
#pragma unroll
                for (int r = 0; r < 8; ++r) {
                    *(f32x4*)(dst + r * 1024) = f32x4{acc[0][8 + r], acc[1][8 + r], acc[2][8 + r], acc[3][8 + r]};
                    v[r] = f32x4{acc[0][r], acc[1][r], acc[2][r], acc[3][r]};
                }
            } else {
#pragma unroll
                for (int r = 0; r < 8; ++r) {
                    *(f32x4*)(dst + r * 1024) = f32x4{acc[0][r], acc[1][r], acc[2][r], acc[3][r]};
                    v[r] = f32x4{acc[0][8 + r], acc[1][8 + r], acc[2][8 + r], acc[3][8 + r]};
                }
            }
        }
        const bool bad = __builtin_amdgcn_readfirstlane(ld_agt(tmo)) == call;
        __syncthreads();
        char* const reg = lds + wv * REG_B;
#pragma unroll
        for (int r = 0; r < 8; ++r) v[r] += *(const f32x4*)(reg + r * 1024 + lane * 16);
        if (bad) {
#pragma unroll
            for (int r = 0; r < 8; ++r) v[r] = f32x4{__builtin_nanf(""), __builtin_nanf(""), __builtin_nanf(""), __builtin_nanf("")};
        }
        asm volatile("s_waitcnt lgkmcnt(0)" ::: "memory");
        unsigned off = 0, bits = 0;
#pragma unroll
        for (int r = 0; r < 8; ++r)
#pragma unroll
            for (int c = 0; c < 4; ++c) {
                const float val = v[r][c];
                const bool keep = !(val <= 0.f);
                const u64 m = __builtin_amdgcn_ballot_w64(keep);
                if (keep) *(float*)(reg + 256 + (off + mbcnt64(m)) * 4) = val;
                v[r][c] = keep ? val : 0.f;
                bits |= keep ? (1u << (r * 4 + c)) : 0u;
                off += __builtin_popcountll(m);
            }
        *(unsigned*)(reg + lane * 4) = bits;
        const unsigned n16 = 16 + ((off + 3) >> 2);
        u32x4 pay[9];
#pragma unroll
        for (int it = 0; it < 9; ++it)
            if (it * 64 < n16) pay[it] = *(const u32x4*)(reg + ((it * 64 + lane) < 544 ? (it * 64 + lane) : 0) * 16);
        STAMP(3);
        const int strip = (stage * 64 + rb * 8 + (cb & 7)) * 8 + wm * 2 + kh;
        const size_t soff = OFF_O + ((size_t)rank * 1024 + strip) * SLOT_B;
        for (int i = 0; i < 7; ++i) {
            const int d = (rank + 1 + (strip + i) % 7) & 7;
            char* dst = p.ws[d] + soff;
#pragma unroll
            for (int it = 0; it < 9; ++it)
                if (it * 64 + lane < n16) st16_sys(dst + 128 + (it * 64 + lane) * 16, pay[it]);
            vm_wait0();
            if (lane == 0) st_sys(dst, (call << 10) | n16);
        }
        STAMP(4);
        float* o = p.out + (size_t)((rank >> 1) * 1024 + rb * 128 + wm * 32 + kh * 16 + (lane >> 5) * 4) * NN
                   + (rank & 1) * 2048 + cb * 128 + (lane & 31) * 4;
#pragma unroll
        for (int r = 0; r < 8; ++r) st16_out(o + (size_t)((r & 3) + 8 * (r >> 2)) * NN, v[r]);
        STAMP(5);
        return;
    }

    const int sw = (u - 16) * 8 + xcd;
    {
        const int g4 = wv >> 2, q4 = wv & 3;
        const int sid = g4 * 128 + sw;
        if (sid < NSEND) {
            unsigned round = 0;
            unsigned* const cnt = (unsigned*)(lds + LDS_CNT);
            f32x4 f[8];
            auto load = [&](const Blk& b) {
                if (b.isw) {
                    const float* src = p.w + (size_t)(b.t * 64 + (lane >> 5) * 32 + q4 * 8) * NN
                                       + b.grp * 2048 + b.lb * 128 + (lane & 31) * 4;
#pragma unroll
                    for (int j = 0; j < 8; ++j) f[j] = *(const f32x4*)(src + (size_t)j * NN);
                } else {
                    const float* src = p.x + (size_t)(b.grp * 1024 + b.lb * 128 + q4 * 32 + (lane & 31)) * KS
                                       + b.t * 64 + (lane >> 5) * 32;
#pragma unroll
                    for (int j = 0; j < 8; ++j) f[j] = *(const f32x4*)(src + j * 4);
                }
            };
            Blk b = decode_blk(sid);
            load(b);
            for (int v = sid; v < 512; v += NSEND) {
                u32x4 h[4];
                if (b.isw) {
#pragma unroll
                    for (int c = 0; c < 4; ++c)
                        h[c] = cvt8(f32x4{f[0][c], f[1][c], f[2][c], f[3][c]}, f32x4{f[4][c], f[5][c], f[6][c], f[7][c]});
                } else {
#pragma unroll
                    for (int q = 0; q < 4; ++q) h[q] = cvt8(f[2 * q], f[2 * q + 1]);
                }
                const int kb = b.t * 8 + rank;
                const size_t doff = (b.isw ? OFF_W : OFF_X) + ((size_t)b.lb * 64 + kb) * 16384 + q4 * 4096 + lane * 16;
                const size_t foff = (b.isw ? OFF_WF : OFF_XF) + b.lb * 256 + kb * 4;
                const int nd = b.isw ? 4 : 2;
                for (int i = 0; i < nd; ++i) {
                    const int ii = (i + b.lb) & (nd - 1);
                    const int d = (b.isw ? b.grp + 2 * ii : b.grp * 2 + ii) & 7;
                    char* dst = p.ws[d] + doff;
#pragma unroll
                    for (int c = 0; c < 4; ++c) st16_sys(dst + c * 1024, h[c]);
                }
                const Blk cur = b;
                if (v + NSEND < 512) { b = decode_blk(v + NSEND); load(b); }
                vm_wait0();
                unsigned old = 0;
                if (lane == 0)
                    old = __hip_atomic_fetch_add(cnt + g4 * 32 + round, 1u,
                                                 __ATOMIC_RELAXED, __HIP_MEMORY_SCOPE_WORKGROUP);
                old = __builtin_amdgcn_readfirstlane(old);
                if (old == 3 && lane < nd) {
                    const int d = (cur.isw ? cur.grp + 2 * lane : cur.grp * 2 + lane) & 7;
                    st_sys(p.ws[d] + foff, call);
                }
                ++round;
                STAMP(round < 8 ? round : 7);
            }
        }
    }

    {
        const int e = wv * 128 + sw;
        char* const reg = lds + wv * REG_B;
        for (int n = 0; n < 7; ++n) {
            const int item = e + n * 1024;
            const int ss = item >> 9;
            const int stage = ss >= 7, slot = ss - stage * 7;
            const int strip = stage * 512 + (item & 511);
            const int s = (rank - 1 - (strip + slot) % 7) & 7;
            const char* src = myws + OFF_O + ((size_t)s * 1024 + strip) * SLOT_B;
            unsigned fl;
            bool bad = false;
            {
                const u64 t0 = __builtin_amdgcn_s_memrealtime();
                while (true) {
                    fl = __builtin_amdgcn_readfirstlane(ld_sys(src));
                    if ((fl >> 10) >= call) break;
                    if (__builtin_amdgcn_readfirstlane(ld_agt(tmo)) == call ||
                        __builtin_amdgcn_s_memrealtime() - t0 > DEADLINE) {
                        if (lane == 0) st_agt(tmo, call);
                        bad = true;
                        break;
                    }
                    __builtin_amdgcn_s_sleep(8);
                }
            }
            f32x4 v[8];
            if (!bad) {
                unsigned n16 = fl & 1023;
                if (n16 > 528) n16 = 528;
                for (unsigned it = 0; it * 64 < n16; ++it)
                    if (it * 64 + lane < n16) glds1k(src + 128 + it * 1024, lane * 16, lds0 + wv * REG_B + it * 1024);
                vm_wait0();
                const unsigned bits = *(const unsigned*)(reg + lane * 4);
                unsigned off = 0;
#pragma unroll
                for (int r = 0; r < 8; ++r)
#pragma unroll
                    for (int c = 0; c < 4; ++c) {
                        const bool keep = (bits >> (r * 4 + c)) & 1;
                        const u64 m = __builtin_amdgcn_ballot_w64(keep);
                        const float val = *(const float*)(reg + 256 + (off + mbcnt64(m)) * 4);
                        v[r][c] = keep ? val : 0.f;
                        off += __builtin_popcountll(m);
                    }
            } else {
#pragma unroll
                for (int r = 0; r < 8; ++r) v[r] = f32x4{__builtin_nanf(""), __builtin_nanf(""), __builtin_nanf(""), __builtin_nanf("")};
            }
            const int tile = strip >> 3, rb = (tile >> 3) & 7, cb = stage * 8 + (tile & 7);
            float* o = p.out + (size_t)((s >> 1) * 1024 + rb * 128 + (strip & 7) * 16 + (lane >> 5) * 4) * NN
                       + (s & 1) * 2048 + cb * 128 + (lane & 31) * 4;
#pragma unroll
            for (int r = 0; r < 8; ++r) st16_out(o + (size_t)((r & 3) + 8 * (r >> 2)) * NN, v[r]);
            STAMP(8 + n);
        }
    }
}

extern "C" void kernel_launch(int rank, int nranks, ncclComm_t comm,
                              hipStream_t stream,
                              void* const* d_in, const int* in_sizes, int n_in,
                              void* d_out, int out_size,
                              const int* in_dtypes, int out_dtype,
                              void* const* d_out_all,
                              void* const* d_ws_all, size_t ws_bytes) {
    if (nranks != NR || (unsigned)rank >= (unsigned)NR || ws_bytes < WS_NEED || n_in < 2 ||
        in_sizes[0] != MM * KS || in_sizes[1] != KS * NN || out_size != MM * NN) {
        fprintf(stderr, "kernel_launch: unexpected problem shape\n");
        return;
    }
    HIP_CHECK(hipFuncSetAttribute((const void*)gemm_ar_kernel, hipFuncAttributeMaxDynamicSharedMemorySize, LDS_BYTES));
    Params p;
    p.x = (const float*)d_in[0];
    p.w = (const float*)d_in[1];
    p.out = (float*)d_out;
    for (int i = 0; i < NR; ++i) p.ws[i] = (char*)d_ws_all[i];
    p.rank = rank;
    hipLaunchKernelGGL(gemm_ar_kernel, dim3(NWG), dim3(NTHR), LDS_BYTES, stream, p);
    HIP_CHECK(hipGetLastError());
}
```

```cpp
#include <hip/hip_runtime.h>
#include <rccl/rccl.h>
#include <cstdio>
#include <cstdint>

#define HIP_CHECK(expr) do { hipError_t _e = (expr); if (_e != hipSuccess) { \
    fprintf(stderr, "HIP error %s at %s:%d\n", hipGetErrorString(_e), __FILE__, __LINE__); \
    return; } } while (0)

typedef unsigned u32x4 __attribute__((ext_vector_type(4)));
typedef float f32x4 __attribute__((ext_vector_type(4)));
typedef float f32x16 __attribute__((ext_vector_type(16)));
typedef _Float16 f16x8 __attribute__((ext_vector_type(8)));
typedef unsigned long long u64;
typedef __attribute__((address_space(1))) unsigned gu32;
typedef __attribute__((address_space(1))) unsigned long long gu64;

constexpr int NR = 8;
constexpr int MM = 4096, NN = 4096, KS = 512;
constexpr int NWG = 256, NTHR = 512;
#define NSEND 64
#define GATE 32
#define OSC "sc1"
#define WT_FROM 5
#define PSLEEP 16
constexpr int NSLOT = 4;
constexpr int PD = NSLOT - 1;
constexpr int REG_B = 9216;
constexpr int EXC_OFF = 8704, EXC_MAX = 128;
constexpr int LDS_BYTES = 132 * 1024;
constexpr int LDS_BAD = 131 * 1024;
constexpr u64 DEADLINE = 40000000ull;

constexpr size_t OFF_SEQ = 0;
constexpr size_t OFF_TMO = 32768;
constexpr size_t OFF_TKT = 32768 + 256;
constexpr size_t OFF_DONE = 32768 + 512;
constexpr size_t OFF_XF = 65536;
constexpr size_t OFF_WF = OFF_XF + 8 * 256;
constexpr size_t OFF_DBG = 262144;
constexpr size_t OFF_I = 2u << 20;
constexpr size_t SRC_B = 3u << 20;
constexpr size_t OFF_O = OFF_I + NR * SRC_B;
constexpr size_t SLOT_B = 8704;
constexpr size_t WS_NEED = OFF_O + (size_t)NR * 1024 * SLOT_B;

struct Params {
    const float* x;
    const float* w;
    float* out;
    char* ws[NR];
    int rank;
};

__device__ __forceinline__ void st16_sys(void* p, u32x4 v) {
    asm volatile("global_store_dwordx4 %0, %1, off sc0 sc1\n\ts_nop 1" :: "v"(p), "v"(v) : "memory");
}
__device__ __forceinline__ void st16_out(void* p, f32x4 v) {
    asm volatile("global_store_dwordx4 %0, %1, off\n\ts_nop 1" :: "v"(p), "v"(v) : "memory");
}
__device__ __forceinline__ void st16_wt(void* p, f32x4 v) {
    asm volatile("global_store_dwordx4 %0, %1, off " OSC "\n\ts_nop 1" :: "v"(p), "v"(v) : "memory");
}
__device__ __forceinline__ void vm_wait0() { asm volatile("s_waitcnt vmcnt(0)" ::: "memory"); }
__device__ __forceinline__ unsigned ld_sys(const void* p) {
    return __hip_atomic_load((gu32*)p, __ATOMIC_RELAXED, __HIP_MEMORY_SCOPE_SYSTEM);
}
__device__ __forceinline__ void st_sys(void* p, unsigned v) {
    __hip_atomic_store((gu32*)p, v, __ATOMIC_RELAXED, __HIP_MEMORY_SCOPE_SYSTEM);
}
__device__ __forceinline__ unsigned ld_agt(const void* p) {
    return __hip_atomic_load((gu32*)p, __ATOMIC_RELAXED, __HIP_MEMORY_SCOPE_AGENT);
}
__device__ __forceinline__ void st_agt(void* p, unsigned v) {
    __hip_atomic_store((gu32*)p, v, __ATOMIC_RELAXED, __HIP_MEMORY_SCOPE_AGENT);
}
#define GSC "sc0 sc1"
__device__ __forceinline__ void glds4k(const char* sbase, unsigned voff, unsigned lds_dst) {
    unsigned keep;
    asm volatile("s_mov_b32 %0, m0\n\ts_mov_b32 m0, %3\n\ts_nop 3\n\t"
                 "global_load_lds_dwordx4 %1, %2 " GSC "\n\t"
                 "global_load_lds_dwordx4 %1, %2 offset:1024 " GSC "\n\t"
                 "global_load_lds_dwordx4 %1, %2 offset:2048 " GSC "\n\t"
                 "global_load_lds_dwordx4 %1, %2 offset:3072 " GSC "\n\t"
                 "s_mov_b32 m0, %0"
                 : "=&s"(keep) : "v"(voff), "s"(sbase), "s"(lds_dst) : "memory");
}
__device__ __forceinline__ void glds1k(const char* sbase, unsigned voff, unsigned lds_dst) {
    unsigned keep;
    asm volatile("s_mov_b32 %0, m0\n\ts_mov_b32 m0, %3\n\ts_nop 3\n\t"
                 "global_load_lds_dwordx4 %1, %2 sc0 sc1\n\t"
                 "s_mov_b32 m0, %0"
                 : "=&s"(keep) : "v"(voff), "s"(sbase), "s"(lds_dst) : "memory");
}
__device__ __forceinline__ unsigned mbcnt64(u64 m) {
    return __builtin_amdgcn_mbcnt_hi((unsigned)(m >> 32), __builtin_amdgcn_mbcnt_lo((unsigned)m, 0u));
}
__device__ __forceinline__ const char* uni_ptr(const char* p) {
    u64 v = (u64)p;
    unsigned lo = __builtin_amdgcn_readfirstlane((unsigned)v), hi = __builtin_amdgcn_readfirstlane((unsigned)(v >> 32));
    return (const char*)(((u64)hi << 32) | lo);
}
__device__ __forceinline__ u32x4 cvt8(f32x4 a, f32x4 b) {
    f16x8 h;
    h[0] = (_Float16)a[0]; h[1] = (_Float16)a[1]; h[2] = (_Float16)a[2]; h[3] = (_Float16)a[3];
    h[4] = (_Float16)b[0]; h[5] = (_Float16)b[1]; h[6] = (_Float16)b[2]; h[7] = (_Float16)b[3];
    return __builtin_bit_cast(u32x4, h);
}

#define STAMP(i) do {} while (0)

struct Blk { int isw, grp, lb, t; };
__device__ __forceinline__ Blk decode_blk(int v) {
    Blk b;
    if (v < 384) {
        b.t = v / 48; int e = v % 48, e3 = e / 3, r3 = e % 3;
        if (r3 == 2) { b.isw = 1; b.grp = e3 & 1; b.lb = e3 >> 1; }
        else { int idx = e3 * 2 + r3; b.isw = 0; b.grp = idx & 3; b.lb = idx >> 2; }
    } else {
        int u = v - 384; b.t = u >> 4; int e = u & 15;
        b.isw = 1; b.grp = e & 1; b.lb = 8 + (e >> 1);
    }
    return b;
}

__global__ void __launch_bounds__(NTHR, 2) gemm_ar_kernel(Params p) {
    extern __shared__ __attribute__((aligned(16))) char lds[];
    const int tid = threadIdx.x, lane = tid & 63;
    const int wv = __builtin_amdgcn_readfirstlane(tid >> 6);
    const int rank = p.rank;
    char* const myws = p.ws[rank];
    const int xcd = blockIdx.x & 7, u = blockIdx.x >> 3;
    const unsigned lds0 = (unsigned)(uintptr_t)(__attribute__((address_space(3))) char*)lds;
    char* const tmo = myws + OFF_TMO;
    STAMP(0);

    const unsigned call = __builtin_amdgcn_readfirstlane(ld_agt(myws + OFF_SEQ + blockIdx.x * 128)) + 1;
    __syncthreads();
    if (tid == 0) st_agt(myws + OFF_SEQ + blockIdx.x * 128, call);

    if (u < 16) {
        const int stage = u >> 3, ul = u & 7;
        const int rb = (xcd >> 1) * 2 + (ul >> 2);
        const int cb = stage * 8 + (xcd & 1) * 4 + (ul & 3);
        const int wm = wv & 3, kh = wv >> 2;
        const char* const gsrc = myws + OFF_I + (size_t)(wv < 4 ? rb : 8 + cb) * (8 * 16384);
        const unsigned voff = (wv & 3) * 4096 + lane * 16;
        const int fpos = (lane & ~7) | ((rank + lane) & 7);
        const char* const xf = myws + OFF_XF + rb * 256 + fpos * 4;
        const char* const wf = myws + OFF_WF + cb * 256 + fpos * 4;
        u64 ready = 0;
        unsigned timed_out = 0;
        auto ensure = [&](int n) {
            if ((ready >> n) & 1) return;
            const u64 t0 = __builtin_amdgcn_s_memrealtime();
            for (unsigned spin = 1;; ++spin) {
                unsigned a = ld_sys(xf), b = ld_sys(wf);
                ready = __builtin_amdgcn_ballot_w64(a >= call && b >= call);
                if ((ready >> n) & 1) break;
                if ((spin & 255) == 0 && (__builtin_amdgcn_readfirstlane(ld_agt(tmo)) == call ||
                                          __builtin_amdgcn_s_memrealtime() - t0 > DEADLINE)) {
                    if (lane == 0) st_agt(tmo, call);
                    vm_wait0();
                    ready = ~0ull;
                    timed_out = 1;
                    break;
                }
                __builtin_amdgcn_s_sleep(8);
            }
        };
        auto issue = [&](int n) {
            glds4k(gsrc + (size_t)((rank + n) & 7) * SRC_B + (n >> 3) * 16384, voff, lds0 + (n % NSLOT) * 32768 + wv * 4096);
        };
        f32x16 acc[4];
#pragma unroll
        for (int c = 0; c < 4; ++c)
#pragma unroll
            for (int r = 0; r < 16; ++r) acc[c][r] = 0.f;

        __builtin_amdgcn_s_waitcnt(0);
        if (wv == 0) { for (int n = 0; n < PD; ++n) ensure(n); }
        __builtin_amdgcn_s_barrier();
        asm volatile("" ::: "memory");
        STAMP(1);
        for (int n = 0; n < PD; ++n) issue(n);
        for (int n = 0; n < 64; ++n) {
            if (wv == 0 && n + PD < 64) ensure(n + PD);
            if (n + PD <= 64) asm volatile("s_waitcnt vmcnt(%0)" :: "i"(4 * (PD - 1)) : "memory");
            else vm_wait0();
            __builtin_amdgcn_s_barrier();
            asm volatile("" ::: "memory");
            if (n + PD < 64) issue(n + PD);
            const char* sa = lds + (n % NSLOT) * 32768 + lane * 16;
            f16x8 fa[2], fb[2][4];
#pragma unroll
            for (int qq = 0; qq < 2; ++qq) {
                fa[qq] = *(const f16x8*)(sa + (wm * 4 + kh * 2 + qq) * 1024);
#pragma unroll
                for (int c = 0; c < 4; ++c) fb[qq][c] = *(const f16x8*)(sa + 16384 + ((kh * 2 + qq) * 4 + c) * 1024);
            }
            __builtin_amdgcn_sched_barrier(0);
#pragma unroll
            for (int qq = 0; qq < 2; ++qq)
#pragma unroll
                for (int c = 0; c < 4; ++c)
                    acc[c] = __builtin_amdgcn_mfma_f32_32x32x16_f16(fa[qq], fb[qq][c], acc[c], 0, 0, 0);
        }
        STAMP(2);
        __syncthreads();
        f32x4 v[8];
        {
            char* dst = lds + (wv ^ 4) * REG_B + lane * 16;
            if (kh == 0) {
#pragma unroll
                for (int r = 0; r < 8; ++r) {
                    *(f32x4*)(dst + r * 1024) = f32x4{acc[0][8 + r], acc[1][8 + r], acc[2][8 + r], acc[3][8 + r]};
                    v[r] = f32x4{acc[0][r], acc[1][r], acc[2][r], acc[3][r]};
                }
            } else {
#pragma unroll
                for (int r = 0; r < 8; ++r) {
                    *(f32x4*)(dst + r * 1024) = f32x4{acc[0][r], acc[1][r], acc[2][r], acc[3][r]};
                    v[r] = f32x4{acc[0][8 + r], acc[1][8 + r], acc[2][8 + r], acc[3][8 + r]};
                }
            }
        }
        if (tid == 0) *(unsigned*)(lds + LDS_BAD) = timed_out;
        __syncthreads();
        const bool bad = __builtin_amdgcn_readfirstlane(*(const unsigned*)(lds + LDS_BAD)) != 0;
        char* const reg = lds + wv * REG_B;
#pragma unroll
        for (int r = 0; r < 8; ++r) v[r] += *(const f32x4*)(reg + r * 1024 + lane * 16);
        if (bad) {
#pragma unroll
            for (int r = 0; r < 8; ++r) v[r] = f32x4{__builtin_nanf(""), __builtin_nanf(""), __builtin_nanf(""), __builtin_nanf("")};
        }
        asm volatile("s_waitcnt lgkmcnt(0)" ::: "memory");
        unsigned off = 0, bits = 0, vmax = 0;
#pragma unroll
        for (int r = 0; r < 8; ++r)
#pragma unroll
            for (int c = 0; c < 4; ++c) {
                const float val = v[r][c];
                const bool keep = !(val <= 0.f);
                const u64 m = __builtin_amdgcn_ballot_w64(keep);
                if (keep) *(float*)(reg + 256 + (off + mbcnt64(m)) * 4) = val;
                const float kept = keep ? val : 0.f;
                v[r][c] = kept;
                vmax = max(vmax, __float_as_uint(kept));
                bits |= keep ? (1u << (r * 4 + c)) : 0u;
                off += __builtin_popcountll(m);
            }
        *(unsigned*)(reg + lane * 4) = bits;
#pragma unroll
        for (int o = 32; o > 0; o >>= 1) vmax = max(vmax, (unsigned)__shfl_xor((int)vmax, o));
        vmax = __builtin_amdgcn_readfirstlane(vmax);
        const unsigned elo = max((int)(vmax >> 23) - 7, 0);
        const unsigned L = (off + 15) >> 4;
        unsigned ne = 0;
        bool raw = vmax >= 0x7f800000u;
        unsigned* const dv = (unsigned*)(reg + 256);
        if (!raw) {
            for (unsigned p0 = 0; p0 < off; p0 += 256) {
                const u32x4 q = *(const u32x4*)(dv + p0 + lane * 4);
#pragma unroll
                for (int j = 0; j < 4; ++j) {
                    const unsigned idx = p0 + lane * 4 + j, dl = elo - 1 - (q[j] >> 23);
                    const bool ex = idx < off && q[j] < (elo << 23);
                    const u64 m = __builtin_amdgcn_ballot_w64(ex);
                    if (m) {
                        const unsigned pos = ne + mbcnt64(m);
                        if (ex && pos < EXC_MAX) *(unsigned short*)(reg + EXC_OFF + pos * 2) = (unsigned short)(idx | (dl << 11));
                        ne += __builtin_popcountll(m);
                        if (__builtin_amdgcn_ballot_w64(ex && dl > 31)) raw = true;
                    }
                }
            }
            if (ne > EXC_MAX) raw = true;
        }
        unsigned n16;
        if (!raw) {
            for (unsigned pp = lane; pp < L; pp += 64) {
                unsigned t[16];
#pragma unroll
                for (int i = 0; i < 16; ++i) {
                    const unsigned x = dv[i * L + pp];
                    t[i] = (x < (elo << 23) ? (x & 0x7fffffu) : x - (elo << 23)) & 0x3ffffffu;
                }
#pragma unroll
                for (int i = 0; i < 12; ++i) dv[i * L + pp] = t[i] | ((t[13 + (i >> 2)] >> (6 * (i & 3))) << 26);
                dv[12 * L + pp] = t[12] | ((t[13] >> 24) << 26) | ((t[14] >> 24) << 28) | ((t[15] >> 24) << 30);
            }
            dv[13 * L + lane] = *(const unsigned*)(reg + EXC_OFF + lane * 4);
            n16 = 16 + ((13 * L * 4 + ne * 2 + 15) >> 4);
        } else {
            ne = 0;
            n16 = 16 + 4 * L;
        }
        const u64 flagv = ((u64)call << 32) | L | (ne << 8) | (elo << 16) | ((unsigned)raw << 24);
        u32x4 pay[9];
#pragma unroll
        for (int it = 0; it < 9; ++it)
            if (it * 64 < n16) pay[it] = *(const u32x4*)(reg + (it * 64 + lane) * 16);
        STAMP(3);
        if (stage == 0) {
            if (wv == 0) {
                const u64 t0 = __builtin_amdgcn_s_memrealtime();
                while ((int)(__builtin_amdgcn_readfirstlane(ld_agt(myws + OFF_DONE)) - ((call - 1) * NSEND + GATE)) < 0 &&
                       __builtin_amdgcn_s_memrealtime() - t0 < DEADLINE)
                    __builtin_amdgcn_s_sleep(8);
            }
            __builtin_amdgcn_s_barrier();
        }
        STAMP(6);
        const int strip = (stage * 64 + rb * 8 + (cb & 7)) * 8 + wm * 2 + kh;
        const size_t soff = OFF_O + ((size_t)rank * 1024 + strip) * SLOT_B;
        for (int i = 0; i < 7; ++i) {
            const int d = (rank + 1 + (strip + i) % 7) & 7;
            char* dst = p.ws[d] + soff;
#pragma unroll
            for (int it = 0; it < 9; ++it)
                if (it * 64 + lane < n16) st16_sys(dst + 128 + (it * 64 + lane) * 16, pay[it]);
            vm_wait0();
            if (lane == 0) __hip_atomic_store((gu64*)dst, flagv, __ATOMIC_RELAXED, __HIP_MEMORY_SCOPE_SYSTEM);
        }
        STAMP(4);
        float* o = p.out + (size_t)((rank >> 1) * 1024 + rb * 128 + wm * 32 + kh * 16 + (lane >> 5) * 4) * NN
                   + (rank & 1) * 2048 + cb * 128 + (lane & 31) * 4;
#pragma unroll
        for (int r = 0; r < 8; ++r) st16_out(o + (size_t)((r & 3) + 8 * (r >> 2)) * NN, v[r]);
        STAMP(5);
        return;
    }

    const int sw = (u - 16) * 8 + xcd;
    if (wv * 128 + sw < NSEND) {
        f32x4 f[4][8];
        auto load = [&](const Blk& b) {
            if (b.isw) {
                const float* src = p.w + (size_t)(b.t * 64 + (lane >> 5) * 32) * NN
                                   + b.grp * 2048 + b.lb * 128 + (lane & 31) * 4;
#pragma unroll
                for (int q = 0; q < 4; ++q)
#pragma unroll
                    for (int j = 0; j < 8; ++j) f[q][j] = *(const f32x4*)(src + (size_t)(q * 8 + j) * NN);
            } else {
                const float* src = p.x + (size_t)(b.grp * 1024 + b.lb * 128 + (lane & 31)) * KS
                                   + b.t * 64 + (lane >> 5) * 32;
#pragma unroll
                for (int g = 0; g < 4; ++g)
#pragma unroll
                    for (int j = 0; j < 8; ++j) f[g][j] = *(const f32x4*)(src + (size_t)g * 32 * KS + j * 4);
            }
        };
        auto ticket = [&]() -> unsigned {
            unsigned t = 0;
            if (lane == 0) t = __hip_atomic_fetch_add((gu32*)(myws + OFF_TKT), 1u, __ATOMIC_RELAXED, __HIP_MEMORY_SCOPE_AGENT);
            return t;
        };
        unsigned v = wv * 128 + sw, round = 0;
        Blk b = decode_blk(v);
        load(b);
        unsigned tk = ticket();
        while (true) {
            const int kb = b.t * 8 + rank;
            const size_t doff = OFF_I + rank * SRC_B + (size_t)((b.isw * 8 + b.lb) * 8 + b.t) * 16384 + lane * 16;
            const size_t foff = (b.isw ? OFF_WF : OFF_XF) + b.lb * 256 + kb * 4;
            const int nd = b.isw ? 4 : 2;
#pragma unroll
            for (int g = 0; g < 4; ++g) {
                u32x4 h[4];
                if (b.isw) {
#pragma unroll
                    for (int c = 0; c < 4; ++c)
                        h[c] = cvt8(f32x4{f[g][0][c], f[g][1][c], f[g][2][c], f[g][3][c]},
                                    f32x4{f[g][4][c], f[g][5][c], f[g][6][c], f[g][7][c]});
                } else {
#pragma unroll
                    for (int q = 0; q < 4; ++q) h[q] = cvt8(f[g][2 * q], f[g][2 * q + 1]);
                }
                for (int i = 0; i < nd; ++i) {
                    const int ii = (i + b.lb + g) & (nd - 1);
                    const int d = (b.isw ? b.grp + 2 * ii : b.grp * 2 + ii) & 7;
                    char* dst = p.ws[d] + doff + g * 4096;
#pragma unroll
                    for (int c = 0; c < 4; ++c) st16_sys(dst + c * 1024, h[c]);
                }
            }
            const Blk cur = b;
            v = __builtin_amdgcn_readfirstlane(tk) - (call - 1) * 512u + NSEND;
            if (v < 512) { b = decode_blk(v); load(b); tk = ticket(); }
            vm_wait0();
            if (lane < nd) {
                const int d = (cur.isw ? cur.grp + 2 * lane : cur.grp * 2 + lane) & 7;
                st_sys(p.ws[d] + foff, call);
            }
            ++round;
            if (v >= 512) break;
        }
        if (lane == 0) __hip_atomic_fetch_add((gu32*)(myws + OFF_DONE), 1u, __ATOMIC_RELAXED, __HIP_MEMORY_SCOPE_AGENT);
    }

    {
        __builtin_amdgcn_s_waitcnt(0);
        const int e = wv * 128 + sw;
        char* const reg = lds + wv * REG_B;
        int istrip, isrc;
        {
            const int item = e + (lane < 7 ? lane : 0) * 1024;
            const int ss = item >> 9;
            const int st = ss >= 7, slot = ss - st * 7;
            istrip = st * 512 + (item & 511);
            isrc = (rank - 1 - (istrip + slot) % 7) & 7;
        }
        const char* const fp = myws + OFF_O + ((size_t)isrc * 1024 + istrip) * SLOT_B;
        unsigned todo = 0x7f;
        bool bad = false;
        for (int n = 0; n < 7; ++n) {
            u64 fl = 0;
            unsigned rdy = 0;
            if (!bad) {
                const u64 t0 = __builtin_amdgcn_s_memrealtime();
                for (unsigned spin = 1;; ++spin) {
                    if ((todo >> (lane & 31)) & (lane < 7))
                        fl = __hip_atomic_load((gu64*)fp, __ATOMIC_RELAXED, __HIP_MEMORY_SCOPE_SYSTEM);
                    rdy = (unsigned)__builtin_amdgcn_ballot_w64((unsigned)(fl >> 32) >= call) & todo;
                    if (rdy) break;
                    if ((spin & 255) == 0 && (__builtin_amdgcn_readfirstlane(ld_agt(tmo)) == call ||
                                              __builtin_amdgcn_s_memrealtime() - t0 > DEADLINE)) {
                        if (lane == 0) st_agt(tmo, call);
                        bad = true;
                        break;
                    }
                    __builtin_amdgcn_s_sleep(PSLEEP);
                }
            }
            const int pick = __builtin_ctz(bad ? todo : rdy);
            todo &= ~(1u << pick);
            const int strip = __builtin_amdgcn_readlane(istrip, pick), s = __builtin_amdgcn_readlane(isrc, pick) & 7;
            f32x4 v[8];
            if (!bad) {
                const unsigned meta = __builtin_amdgcn_readlane((unsigned)fl, pick);
                const unsigned L = min(meta & 255u, 128u), ne = min((meta >> 8) & 255u, (unsigned)EXC_MAX), elo = (meta >> 16) & 255u;
                const bool raw = (meta >> 24) & 1;
                const unsigned n16 = 16 + (raw ? 4 * L : (13 * L * 4 + ne * 2 + 15) >> 4);
                const char* src = myws + OFF_O + ((size_t)s * 1024 + strip) * SLOT_B + 128;
                for (unsigned it = 0; it * 64 < n16; ++it)
                    if (it * 64 + lane < n16) glds1k(src + it * 1024, lane * 16, lds0 + wv * REG_B + it * 1024);
                vm_wait0();
                unsigned* const dv = (unsigned*)(reg + 256);
                if (!raw) {
                    const unsigned ex = dv[13 * L + lane];
                    for (unsigned pp = lane; pp < L; pp += 64) {
                        unsigned t[13];
#pragma unroll
                        for (int i = 0; i < 13; ++i) t[i] = dv[i * L + pp];
#pragma unroll
                        for (int i = 0; i < 13; ++i) dv[i * L + pp] = (t[i] & 0x3ffffffu) + (elo << 23);
#pragma unroll
                        for (int k = 0; k < 3; ++k)
                            dv[(13 + k) * L + pp] = ((t[4 * k] >> 26) | ((t[4 * k + 1] >> 26) << 6) | ((t[4 * k + 2] >> 26) << 12) |
                                                     ((t[4 * k + 3] >> 26) << 18) | (((t[12] >> (26 + 2 * k)) & 3u) << 24)) + (elo << 23);
                    }
                    if (lane * 2 < ne) dv[ex & 2047u] -= (((ex >> 11) & 31u) + 1) << 23;
                    if (lane * 2 + 1 < ne) dv[(ex >> 16) & 2047u] -= ((ex >> 27) + 1) << 23;
                }
                const unsigned bits = *(const unsigned*)(reg + lane * 4);
                unsigned off = 0;
#pragma unroll
                for (int r = 0; r < 8; ++r)
#pragma unroll
                    for (int c = 0; c < 4; ++c) {
                        const bool keep = (bits >> (r * 4 + c)) & 1;
                        const u64 m = __builtin_amdgcn_ballot_w64(keep);
                        const float val = *(const float*)(reg + 256 + (off + mbcnt64(m)) * 4);
                        v[r][c] = keep ? val : 0.f;
                        off += __builtin_popcountll(m);
                    }
            } else {
#pragma unroll
                for (int r = 0; r < 8; ++r) v[r] = f32x4{__builtin_nanf(""), __builtin_nanf(""), __builtin_nanf(""), __builtin_nanf("")};
            }
            const int tile = strip >> 3, rb = (tile >> 3) & 7, cb = (tile >> 6) * 8 + (tile & 7);
            float* o = p.out + (size_t)((s >> 1) * 1024 + rb * 128 + (strip & 7) * 16 + (lane >> 5) * 4) * NN
                       + (s & 1) * 2048 + cb * 128 + (lane & 31) * 4;
            if (n < WT_FROM) {
#pragma unroll
                for (int r = 0; r < 8; ++r) st16_out(o + (size_t)((r & 3) + 8 * (r >> 2)) * NN, v[r]);
            } else {
#pragma unroll
                for (int r = 0; r < 8; ++r) st16_wt(o + (size_t)((r & 3) + 8 * (r >> 2)) * NN, v[r]);
            }
            STAMP(8 + n);
        }
    }
}

extern "C" void kernel_launch(int rank, int nranks, ncclComm_t comm,
                              hipStream_t stream,
                              void* const* d_in, const int* in_sizes, int n_in,
                              void* d_out, int out_size,
                              const int* in_dtypes, int out_dtype,
                              void* const* d_out_all,
                              void* const* d_ws_all, size_t ws_bytes) {
    if (nranks != NR || (unsigned)rank >= (unsigned)NR || ws_bytes < WS_NEED || n_in < 2 ||
        in_sizes[0] != MM * KS || in_sizes[1] != KS * NN || out_size != MM * NN) {
        fprintf(stderr, "kernel_launch: unexpected problem shape\n");
        return;
    }
    HIP_CHECK(hipFuncSetAttribute((const void*)gemm_ar_kernel, hipFuncAttributeMaxDynamicSharedMemorySize, LDS_BYTES));
    Params p;
    p.x = (const float*)d_in[0];
    p.w = (const float*)d_in[1];
    p.out = (float*)d_out;
    for (int i = 0; i < NR; ++i) p.ws[i] = (char*)d_ws_all[i];
    p.rank = rank;
    hipLaunchKernelGGL(gemm_ar_kernel, dim3(NWG), dim3(NTHR), LDS_BYTES, stream, p);
    HIP_CHECK(hipGetLastError());
}
```

```cpp
#include <hip/hip_runtime.h>
#include <rccl/rccl.h>
#include <cstdio>
#include <cstdint>

#define HIP_CHECK(expr) do { hipError_t _e = (expr); if (_e != hipSuccess) { \
    fprintf(stderr, "HIP error %s at %s:%d\n", hipGetErrorString(_e), __FILE__, __LINE__); \
    return; } } while (0)

typedef unsigned u32x4 __attribute__((ext_vector_type(4)));
typedef float f32x4 __attribute__((ext_vector_type(4)));
typedef float f32x16 __attribute__((ext_vector_type(16)));
typedef _Float16 f16x8 __attribute__((ext_vector_type(8)));
typedef unsigned long long u64;
typedef __attribute__((address_space(1))) unsigned gu32;
typedef __attribute__((address_space(1))) unsigned long long gu64;

constexpr int NR = 8;
constexpr int MM = 4096, NN = 4096, KS = 512;
constexpr int NWG = 256, NTHR = 512;
#define EW 7
#define TOK 2
#define XP (xcd)
#define HOLD 0
#define HOLD2 0
#define TSW (NIN + 512)
#define PSLEEP 8
#define STAG 8
constexpr int NSLOT = 5;
constexpr int REG_B = 9216;
constexpr int EXC_OFF = 8704, EXC_MAX = 128;
constexpr int LDS_BYTES = NSLOT * 32768;
constexpr int LDS_BAD = 8 * REG_B;
constexpr int LDS_TOK = LDS_BAD + 16;
constexpr u64 DEADLINE = 40000000ull;

constexpr size_t OFF_SEQ = 0;
constexpr size_t OFF_TMO = 32768;
constexpr size_t OFF_ORD = 32768 + 256;
constexpr size_t OFF_ETK = 32768 + 1024;
constexpr size_t OFF_XF = 65536;
constexpr int NIN = 352, NITEM = NIN + 1024;
constexpr size_t OFF_OF = 131072;
constexpr size_t OFF_DBG = 262144;
constexpr size_t OFF_I = 2u << 20;
constexpr size_t KC_B = 24 * 16384;
constexpr size_t OFF_P = OFF_I + 64 * KC_B;
constexpr size_t OFF_O = OFF_P + (8u << 20);
constexpr size_t SLOT_B = 8448;
constexpr size_t WS_NEED = OFF_O + (size_t)1024 * SLOT_B;
static_assert(OFF_OF + NR * NITEM * 8 <= OFF_DBG, "layout");

struct Params {
    const float* x;
    const float* w;
    float* out;
    char* ws[NR];
    int rank;
};

__device__ __forceinline__ void st16_sys(void* p, u32x4 v) {
    asm volatile("global_store_dwordx4 %0, %1, off sc0 sc1\n\ts_nop 1" :: "v"(p), "v"(v) : "memory");
}
#define OSC "sc1 nt"
#define USC "sc0 sc1"
#define PSC "sc0 sc1 nt"
__device__ __forceinline__ void st16_wt(void* p, f32x4 v) {
    asm volatile("global_store_dwordx4 %0, %1, off " OSC "\n\ts_nop 1" :: "v"(p), "v"(v) : "memory");
}
__device__ __forceinline__ void st16_loc(void* p, u32x4 v) {
    asm volatile("global_store_dwordx4 %0, %1, off " USC "\n\ts_nop 1" :: "v"(p), "v"(v) : "memory");
}
__device__ __forceinline__ void cfence() { asm volatile("" ::: "memory"); }
__device__ __forceinline__ void vm_wait0() { asm volatile("s_waitcnt vmcnt(0)" ::: "memory"); }
__device__ __forceinline__ u64 ld_sys64(const void* p) {
    return __hip_atomic_load((gu64*)p, __ATOMIC_RELAXED, __HIP_MEMORY_SCOPE_SYSTEM);
}
__device__ __forceinline__ void st_sys64(void* p, u64 v) {
    __hip_atomic_store((gu64*)p, v, __ATOMIC_RELAXED, __HIP_MEMORY_SCOPE_SYSTEM);
}
__device__ __forceinline__ void st_sys(void* p, unsigned v) {
    __hip_atomic_store((gu32*)p, v, __ATOMIC_RELAXED, __HIP_MEMORY_SCOPE_SYSTEM);
}
__device__ __forceinline__ unsigned ld_agt(const void* p) {
    return __hip_atomic_load((gu32*)p, __ATOMIC_RELAXED, __HIP_MEMORY_SCOPE_AGENT);
}
__device__ __forceinline__ void st_agt(void* p, unsigned v) {
    __hip_atomic_store((gu32*)p, v, __ATOMIC_RELAXED, __HIP_MEMORY_SCOPE_AGENT);
}
#define GSC "sc0 sc1"
__device__ __forceinline__ void glds4k(const char* sbase, unsigned voff, unsigned lds_dst) {
    unsigned keep;
    asm volatile("s_mov_b32 %0, m0\n\ts_mov_b32 m0, %3\n\ts_nop 3\n\t"
                 "global_load_lds_dwordx4 %1, %2 " GSC "\n\t"
                 "global_load_lds_dwordx4 %1, %2 offset:1024 " GSC "\n\t"
                 "global_load_lds_dwordx4 %1, %2 offset:2048 " GSC "\n\t"
                 "global_load_lds_dwordx4 %1, %2 offset:3072 " GSC "\n\t"
                 "s_mov_b32 m0, %0"
                 : "=&s"(keep) : "v"(voff), "s"(sbase), "s"(lds_dst) : "memory");
}
__device__ __forceinline__ void glds1k(const char* sbase, unsigned voff, unsigned lds_dst) {
    unsigned keep;
    asm volatile("s_mov_b32 %0, m0\n\ts_mov_b32 m0, %3\n\ts_nop 3\n\t"
                 "global_load_lds_dwordx4 %1, %2 " PSC "\n\t"
                 "s_mov_b32 m0, %0"
                 : "=&s"(keep) : "v"(voff), "s"(sbase), "s"(lds_dst) : "memory");
}
__device__ __forceinline__ unsigned mbcnt64(u64 m) {
    return __builtin_amdgcn_mbcnt_hi((unsigned)(m >> 32), __builtin_amdgcn_mbcnt_lo((unsigned)m, 0u));
}
__device__ __forceinline__ const char* uni_ptr(const char* p) {
    u64 v = (u64)p;
    unsigned lo = __builtin_amdgcn_readfirstlane((unsigned)v), hi = __builtin_amdgcn_readfirstlane((unsigned)(v >> 32));
    return (const char*)(((u64)hi << 32) | lo);
}
__device__ __forceinline__ u32x4 cvt8(f32x4 a, f32x4 b) {
    f16x8 h;
    h[0] = (_Float16)a[0]; h[1] = (_Float16)a[1]; h[2] = (_Float16)a[2]; h[3] = (_Float16)a[3];
    h[4] = (_Float16)b[0]; h[5] = (_Float16)b[1]; h[6] = (_Float16)b[2]; h[7] = (_Float16)b[3];
    return __builtin_bit_cast(u32x4, h);
}

typedef unsigned short u16x2 __attribute__((ext_vector_type(2)));
__device__ __forceinline__ unsigned pk_subsat(unsigned a, unsigned b) {
    return __builtin_bit_cast(unsigned, __builtin_elementwise_sub_sat(__builtin_bit_cast(u16x2, a), __builtin_bit_cast(u16x2, b)));
}
__device__ __forceinline__ unsigned pk_max(unsigned a, unsigned b) {
    return __builtin_bit_cast(unsigned, __builtin_elementwise_max(__builtin_bit_cast(u16x2, a), __builtin_bit_cast(u16x2, b)));
}
__device__ __forceinline__ unsigned pk_min(unsigned a, unsigned b) {
    return __builtin_bit_cast(unsigned, __builtin_elementwise_min(__builtin_bit_cast(u16x2, a), __builtin_bit_cast(u16x2, b)));
}

#define STAMP(i) do {} while (0)

struct Blk { int isw, grp, lb, t; };
__device__ __forceinline__ Blk decode_blk(int v) {
    Blk b;
    if (v < 384) {
        b.t = v / 48; int e = v % 48, e3 = e / 3, r3 = e % 3;
        if (r3 == 2) { b.isw = 1; b.grp = e3 & 1; b.lb = e3 >> 1; }
        else { int idx = e3 * 2 + r3; b.isw = 0; b.grp = idx & 3; b.lb = idx >> 2; }
    } else {
        int u = v - 384; b.t = u >> 4; int e = u & 15;
        b.isw = 1; b.grp = e & 1; b.lb = 8 + (e >> 1);
    }
    return b;
}

__global__ void __launch_bounds__(NTHR, 2) gemm_ar_kernel(Params p) {
    extern __shared__ __attribute__((aligned(16))) char lds[];
    const int tid = threadIdx.x, lane = tid & 63;
    const int wv = __builtin_amdgcn_readfirstlane(tid >> 6);
    const int rank = p.rank;
    char* const myws = p.ws[rank];
    const int xcd = blockIdx.x & 7, u = blockIdx.x >> 3;
    const unsigned lds0 = (unsigned)(uintptr_t)(__attribute__((address_space(3))) char*)lds;
    char* const tmo = myws + OFF_TMO;
    STAMP(0);

    const unsigned call = __builtin_amdgcn_readfirstlane(ld_agt(myws + OFF_SEQ + blockIdx.x * 128)) + 1;
    if (tid < 2) ((unsigned*)(lds + LDS_TOK))[tid] = 0;
    __syncthreads();
    if (tid == 0) st_agt(myws + OFF_SEQ + blockIdx.x * 128, call);
    STAMP(7);

    if (u < 16) {
        const int stage = u >> 3, ul = u & 7;
        const int rb = (xcd >> 1) * 2 + (ul >> 2);
        const int cb = stage * 8 + (xcd & 1) * 4 + (ul & 3);
        const int wm = wv & 3, kh = wv >> 2;
        const char* const gsrc = myws + OFF_I + (size_t)(wv < 4 ? rb : 8 + cb) * 16384;
        const unsigned voff = (wv & 3) * 4096 + lane * 16;
        const int fpos = (lane & ~7) | ((rank + lane) & 7);
        const char* const xf = myws + OFF_XF + rb * 512 + fpos * 8;
        const char* const wf = myws + OFF_XF + (8 + cb) * 512 + fpos * 8;
        u64 ready = 0;
        unsigned timed_out = 0;
        auto ensure = [&](int n) {
            if ((ready >> n) & 1) return;
            const u64 t0 = __builtin_amdgcn_s_memrealtime();
            for (unsigned spin = 1;; ++spin) {
                const u64 a = ld_sys64(xf), b = ld_sys64(wf);
                ready = __builtin_amdgcn_ballot_w64(min(min((unsigned)a, (unsigned)(a >> 32)), min((unsigned)b, (unsigned)(b >> 32))) >= call);
                if ((ready >> n) & 1) break;
                if ((spin & 255) == 0 && (__builtin_amdgcn_readfirstlane(ld_agt(tmo)) == call ||
                                          __builtin_amdgcn_s_memrealtime() - t0 > DEADLINE)) {
                    if (lane == 0) st_agt(tmo, call);
                    vm_wait0();
                    ready = ~0ull;
                    timed_out = 1;
                    break;
                }
                __builtin_amdgcn_s_sleep(8);
            }
        };
        auto issue = [&](int n, int slot) {
            glds4k(gsrc + (size_t)((n & ~7) | ((rank + n) & 7)) * KC_B, voff, lds0 + slot * 32768 + wv * 4096);
        };
        struct Frag { f16x8 a[2], b[2][4]; };
        auto rd = [&](Frag& f, int slot) {
            const char* sa = lds + slot * 32768 + lane * 16;
#pragma unroll
            for (int qq = 0; qq < 2; ++qq) {
                f.a[qq] = *(const f16x8*)(sa + (wm * 4 + kh * 2 + qq) * 1024);
#pragma unroll
                for (int c = 0; c < 4; ++c) f.b[qq][c] = *(const f16x8*)(sa + 16384 + ((kh * 2 + qq) * 4 + c) * 1024);
            }
        };
        f32x16 acc[4];
#pragma unroll
        for (int c = 0; c < 4; ++c)
#pragma unroll
            for (int r = 0; r < 16; ++r) acc[c][r] = 0.f;

        __builtin_amdgcn_s_waitcnt(0);
        if (wv == 0) { for (int n = 0; n < NSLOT; ++n) ensure(n); }
        __builtin_amdgcn_s_barrier();
        asm volatile("" ::: "memory");
        STAMP(1);
        for (int n = 0; n < NSLOT; ++n) issue(n, n);
        asm volatile("s_waitcnt vmcnt(%0)" :: "i"(4 * (NSLOT - 1)) : "memory");
        __builtin_amdgcn_s_barrier();
        asm volatile("" ::: "memory");
        Frag f0, f1;
        rd(f0, 0);
        int slot = 0;
        auto step = [&](int n, Frag& cur, Frag& nxt) {
            if (wv == 0 && n + NSLOT < 64) ensure(n + NSLOT);
            if (n + NSLOT <= 64) asm volatile("s_waitcnt vmcnt(%0)" :: "i"(4 * (NSLOT - 2)) : "memory");
            else vm_wait0();
            __builtin_amdgcn_s_waitcnt(0xC07F);
            __builtin_amdgcn_s_barrier();
            asm volatile("" ::: "memory");
            const int ns = slot == NSLOT - 1 ? 0 : slot + 1;
            rd(nxt, ns);
            if (n + NSLOT < 64) issue(n + NSLOT, slot);
            slot = ns;
            __builtin_amdgcn_sched_barrier(0);
#pragma unroll
            for (int qq = 0; qq < 2; ++qq)
#pragma unroll
                for (int c = 0; c < 4; ++c)
                    acc[c] = __builtin_amdgcn_mfma_f32_32x32x16_f16(cur.a[qq], cur.b[qq][c], acc[c], 0, 0, 0);
            __builtin_amdgcn_sched_barrier(0);
        };
        for (int n = 0; n < 64; n += 2) { step(n, f0, f1); step(n + 1, f1, f0); }
        STAMP(2);
        unsigned ord = 0;
        if (tid == 0) ord = __hip_atomic_fetch_add((gu32*)(myws + OFF_ORD), 1u, __ATOMIC_RELAXED, __HIP_MEMORY_SCOPE_AGENT);
        __syncthreads();
        f32x4 v[8];
        {
            char* dst = lds + (wv ^ 4) * REG_B + lane * 16;
            if (kh == 0) {
#pragma unroll
                for (int r = 0; r < 8; ++r) {
                    *(f32x4*)(dst + r * 1024) = f32x4{acc[0][8 + r], acc[1][8 + r], acc[2][8 + r], acc[3][8 + r]};
                    v[r] = f32x4{acc[0][r], acc[1][r], acc[2][r], acc[3][r]};
                }
            } else {
#pragma unroll
                for (int r = 0; r < 8; ++r) {
                    *(f32x4*)(dst + r * 1024) = f32x4{acc[0][r], acc[1][r], acc[2][r], acc[3][r]};
                    v[r] = f32x4{acc[0][8 + r], acc[1][8 + r], acc[2][8 + r], acc[3][8 + r]};
                }
            }
        }
        if (tid == 0) *(unsigned*)(lds + LDS_BAD) = (ord & 127) | (timed_out << 7);
        __syncthreads();
        ord = __builtin_amdgcn_readfirstlane(*(const unsigned*)(lds + LDS_BAD));
        const bool bad = ord >> 7;
        ord &= 127;
        char* const reg = lds + wv * REG_B;
#pragma unroll
        for (int r = 0; r < 8; ++r) v[r] += *(const f32x4*)(reg + r * 1024 + lane * 16);
        if (bad) {
#pragma unroll
            for (int r = 0; r < 8; ++r) v[r] = f32x4{__builtin_nanf(""), __builtin_nanf(""), __builtin_nanf(""), __builtin_nanf("")};
        }
        asm volatile("s_waitcnt lgkmcnt(0)" ::: "memory");
        unsigned off = 0, bits = 0, vmax = 0;
#pragma unroll
        for (int r = 0; r < 8; ++r)
#pragma unroll
            for (int c = 0; c < 4; ++c) {
                const float val = v[r][c];
                const bool keep = !(val <= 0.f);
                const u64 m = __builtin_amdgcn_ballot_w64(keep);
                if (keep) *(float*)(reg + 256 + (off + mbcnt64(m)) * 4) = val;
                const float kept = keep ? val : 0.f;
                v[r][c] = kept;
                vmax = max(vmax, __float_as_uint(kept));
                bits |= keep ? (1u << (r * 4 + c)) : 0u;
                off += __builtin_popcountll(m);
            }
        *(unsigned*)(reg + lane * 4) = bits;
        cfence();
#pragma unroll
        for (int o = 32; o > 0; o >>= 1) vmax = max(vmax, (unsigned)__shfl_xor((int)vmax, o));
        vmax = __builtin_amdgcn_readfirstlane(vmax);
        const unsigned elo = max((int)(vmax >> 23) - 7, 0);
        const unsigned L = (off + 15) >> 4;
        unsigned ne = 0;
        bool raw = vmax >= 0x7f800000u;
        unsigned* const dv = (unsigned*)(reg + 256);
        if (!raw) {
            for (unsigned p0 = 0; p0 < off; p0 += 256) {
                const u32x4 q = *(const u32x4*)(dv + p0 + lane * 4);
#pragma unroll
                for (int j = 0; j < 4; ++j) {
                    const unsigned idx = p0 + lane * 4 + j, dl = elo - 1 - (q[j] >> 23);
                    const bool ex = idx < off && q[j] < (elo << 23);
                    const u64 m = __builtin_amdgcn_ballot_w64(ex);
                    if (m) {
                        const unsigned pos = ne + mbcnt64(m);
                        if (ex && pos < EXC_MAX) *(unsigned short*)(reg + EXC_OFF + pos * 2) = (unsigned short)(idx | (dl << 11));
                        ne += __builtin_popcountll(m);
                        if (__builtin_amdgcn_ballot_w64(ex && dl > 31)) raw = true;
                    }
                }
            }
            if (ne > EXC_MAX) raw = true;
        }
        unsigned n16;
        if (!raw) {
            for (unsigned pp = lane; pp < L; pp += 64) {
                unsigned t[16];
#pragma unroll
                for (int i = 0; i < 16; ++i) {
                    const unsigned x = dv[i * L + pp];
                    t[i] = (x < (elo << 23) ? (x & 0x7fffffu) : x - (elo << 23)) & 0x3ffffffu;
                }
#pragma unroll
                for (int i = 0; i < 12; ++i) dv[i * L + pp] = t[i] | ((t[13 + (i >> 2)] >> (6 * (i & 3))) << 26);
                dv[12 * L + pp] = t[12] | ((t[13] >> 24) << 26) | ((t[14] >> 24) << 28) | ((t[15] >> 24) << 30);
            }
            cfence();
            dv[13 * L + lane] = *(const unsigned*)(reg + EXC_OFF + lane * 4);
            n16 = 16 + ((13 * L * 4 + ne * 2 + 15) >> 4);
        } else {
            ne = 0;
            n16 = 16 + 4 * L;
        }
        cfence();
        const int tile = stage * 64 + rb * 8 + (cb & 7), ts = wm * 2 + kh;
        const u64 flagv = ((u64)call << 32) | L | (ne << 8) | (elo << 16) | ((unsigned)raw << 24) | ((unsigned)tile << 25);
        {
            char* dst = myws + OFF_O + (size_t)(tile * 8 + ts) * SLOT_B;
            for (unsigned i = lane; i < n16; i += 64) st16_sys(dst + i * 16, *(const u32x4*)(reg + i * 16));
        }
        STAMP(3);
        vm_wait0();
        if (lane < NR && lane != rank) st_sys64(p.ws[lane] + OFF_OF + (rank * NITEM + NIN + ord * 8 + ts) * 8, flagv);
        STAMP(4);
        float* o = p.out + (size_t)((rank >> 1) * 1024 + rb * 128 + wm * 32 + kh * 16 + (lane >> 5) * 4) * NN
                   + (rank & 1) * 2048 + cb * 128 + (lane & 31) * 4;
#pragma unroll
        for (int r = 0; r < 8; ++r) st16_wt(o + (size_t)((r & 3) + 8 * (r >> 2)) * NN, v[r]);
        STAMP(5);
        return;
    }

    char* const reg = lds + wv * REG_B;
    {
        const int o = wv * 128 + (u - 16) * 8 + xcd, hb = o & 1;
        const Blk b = decode_blk(o >> 1);
        for (int i = 0; i < wv; ++i) __builtin_amdgcn_s_sleep(STAG);
        u32x4 h[8];
        {
            f32x4 f[2][8];
            if (b.isw) {
                const float* src = p.w + (size_t)(b.t * 64 + (lane >> 5) * 32 + hb * 16) * NN + b.grp * 2048 + b.lb * 128 + (lane & 31) * 4;
#pragma unroll
                for (int j = 0; j < 16; ++j) f[j >> 3][j & 7] = *(const f32x4*)(src + (size_t)j * NN);
#pragma unroll
                for (int g = 0; g < 2; ++g)
#pragma unroll
                    for (int c = 0; c < 4; ++c)
                        h[g * 4 + c] = cvt8(f32x4{f[g][0][c], f[g][1][c], f[g][2][c], f[g][3][c]},
                                            f32x4{f[g][4][c], f[g][5][c], f[g][6][c], f[g][7][c]});
            } else {
                const float* src = p.x + (size_t)(b.grp * 1024 + b.lb * 128 + hb * 64 + (lane & 31)) * KS + b.t * 64 + (lane >> 5) * 32;
#pragma unroll
                for (int j = 0; j < 16; ++j) f[j >> 3][j & 7] = *(const f32x4*)(src + (size_t)(j >> 3) * 32 * KS + (j & 7) * 4);
#pragma unroll
                for (int g = 0; g < 2; ++g)
#pragma unroll
                    for (int q = 0; q < 4; ++q) h[g * 4 + q] = cvt8(f[g][2 * q], f[g][2 * q + 1]);
            }
        }
        STAMP(15);
        const int nd = b.isw ? 4 : 2, blk = b.isw * 8 + b.lb;
        const int dl = (b.isw ? b.grp + 2 * lane : b.grp * 2 + lane) & 7;
        const size_t doff = OFF_I + (b.t * 8 + rank) * KC_B + blk * 16384 + hb * 8192 + lane * 16;
        const size_t foff = OFF_XF + blk * 512 + (b.t * 8 + rank) * 8 + hb * 4;
        if (o < 96) {
            for (int i = 0; i < nd; ++i) {
                const int ii = (i + b.lb + hb) & (nd - 1);
                char* dst = p.ws[(b.isw ? b.grp + 2 * ii : b.grp * 2 + ii) & 7] + doff;
#pragma unroll
                for (int c = 0; c < 8; ++c) st16_sys(dst + c * 1024, h[c]);
            }
            vm_wait0();
            if (lane < nd) st_sys(p.ws[dl] + foff, call);
        } else {
            if (b.grp == (b.isw ? rank & 1 : rank >> 1)) {
#pragma unroll
                for (int c = 0; c < 8; ++c) st16_loc(myws + doff + c * 1024, h[c]);
            }
            unsigned mx = 0;
#pragma unroll
            for (int c = 0; c < 8; ++c) {
                *(u32x4*)(reg + c * 1024 + lane * 16) = h[c];
#pragma unroll
                for (int j = 0; j < 4; ++j) mx = pk_max(mx, h[c][j] & 0x7c007c00u);
            }
            mx = max(mx >> 16, mx & 0xffffu);
#pragma unroll
            for (int d = 32; d > 0; d >>= 1) mx = max(mx, (unsigned)__shfl_xor((int)mx, d));
            const unsigned lo = max((int)(__builtin_amdgcn_readfirstlane(mx) >> 10) - 7, 0), base = lo * 0x04000400u;
            u32x4 q[8];
            unsigned em[2] = {0, 0};
#pragma unroll
            for (int c = 0; c < 8; ++c)
#pragma unroll
                for (int j = 0; j < 4; ++j) {
                    const unsigned wd = h[c][j], ea = wd & 0x7c007c00u;
                    q[c][j] = (wd & 0x83ff83ffu) | pk_subsat(ea, base);
                    em[c >> 2] |= pk_min(pk_subsat(base, ea), 0x00010001u) << ((c & 3) * 4 + j);
                }
            u64 m = ((u64)em[1] << 32) | em[0];
            cfence();
            unsigned ne = 0;
            bool raw = false;
            for (u64 bal; (bal = __builtin_amdgcn_ballot_w64(m != 0)) != 0 && !raw;) {
                bool far = false;
                if (m != 0) {
                    const unsigned bt = __builtin_ctzll(m), wi = (bt & 15) + (bt >> 5) * 16;
                    m &= m - 1;
                    const unsigned idx = (wi >> 2) * 512 + lane * 8 + (wi & 3) * 2 + ((bt >> 4) & 1);
                    const unsigned dlt = lo - 1 - ((*(const unsigned short*)(reg + idx * 2) >> 10) & 31u), pos = ne + mbcnt64(bal);
                    far = dlt > 15;
                    if (pos < 256) *(unsigned short*)(reg + EXC_OFF + pos * 2) = (unsigned short)(idx | (dlt << 12));
                }
                ne += __builtin_popcountll(bal);
                raw = ne > 256 || __builtin_amdgcn_ballot_w64(far) != 0;
            }
            cfence();
            char* dst = myws + OFF_P + (size_t)((b.t * 64 + (b.isw ? 32 + b.grp * 16 : b.grp * 8) + b.lb) * 2 + hb) * 8192 + lane * 16;
            if (raw) {
                ne = 0;
#pragma unroll
                for (int c = 0; c < 8; ++c) st16_sys(dst + c * 1024, h[c]);
            } else {
                const u32x4 top = (q[7] & 0x1fff1fffu) | ((q[7] >> 2) & 0x20002000u);
#pragma unroll
                for (int c = 0; c < 7; ++c) st16_sys(dst + c * 1024, q[c] | ((top << (13 - 2 * c)) & 0x60006000u));
                if (lane * 8 < ne) st16_sys(dst + 7168, *(const u32x4*)(reg + EXC_OFF + lane * 16));
            }
            vm_wait0();
            const int item = b.lb < 8 ? (b.t - 1) * 32 + (b.lb * 2 + b.isw) * 2 + hb : 224 + b.t * 16 + (b.lb - 8) * 2 + hb;
            if (lane < nd) {
                if (dl == rank) st_sys(myws + foff, call);
                else st_sys64(p.ws[dl] + OFF_OF + (rank * NITEM + item) * 8, ((u64)call << 32) | ne | (lo << 16) | ((unsigned)raw << 24));
            }
        }
        STAMP(1);
    }

    {
        __builtin_amdgcn_s_waitcnt(0);
        const int s = (XP) & 7;
        if (s == rank || wv >= EW) return;
        unsigned* const tok = (unsigned*)(lds + LDS_TOK);
        bool bad = false;
        unsigned tk = 0;
        auto ticket = [&]() {
            if (lane == 0) tk = __hip_atomic_fetch_add((gu32*)(myws + OFF_ETK + xcd * 128), 1u, __ATOMIC_RELAXED, __HIP_MEMORY_SCOPE_AGENT);
        };
        auto pull = [&](const char* src, unsigned n16, unsigned wgt, bool hold) {
            unsigned my = 0;
            if (lane == 0) my = __hip_atomic_fetch_add(tok, wgt, __ATOMIC_RELAXED, __HIP_MEMORY_SCOPE_WORKGROUP);
            my = __builtin_amdgcn_readfirstlane(my) + wgt;
            while ((int)(my - __builtin_amdgcn_readfirstlane(__hip_atomic_load(tok + 1, __ATOMIC_RELAXED, __HIP_MEMORY_SCOPE_WORKGROUP))) > TOK)
                __builtin_amdgcn_s_sleep(1);
            if (hold) __builtin_amdgcn_s_sleep(HOLD); else __builtin_amdgcn_s_sleep(HOLD2);
            src = uni_ptr(src);
            for (unsigned it = 0; it * 64 < n16; ++it)
                if (it * 64 + lane < n16) glds1k(src + it * 1024, lane * 16, lds0 + wv * REG_B + it * 1024);
            vm_wait0();
            if (lane == 0) __hip_atomic_fetch_add(tok + 1, wgt, __ATOMIC_RELAXED, __HIP_MEMORY_SCOPE_WORKGROUP);
        };
        ticket();
        for (int n = 0;; ++n) {
            const unsigned k = __builtin_amdgcn_readfirstlane(tk) - (call - 1) * (NITEM + 16 * EW);
            if (k >= NITEM) break;
            const char* const fp = myws + OFF_OF + (s * NITEM + k) * 8;
            unsigned meta = ((k - NIN) >> 3) << 25;
            if (!bad) {
                const u64 t0 = __builtin_amdgcn_s_memrealtime();
                for (unsigned spin = 1;; ++spin) {
                    const u64 fl = ld_sys64(fp);
                    if (__builtin_amdgcn_readfirstlane((unsigned)(fl >> 32)) >= call) { meta = __builtin_amdgcn_readfirstlane((unsigned)fl); break; }
                    if ((spin & 255) == 0 && (__builtin_amdgcn_readfirstlane(ld_agt(tmo)) == call ||
                                              __builtin_amdgcn_s_memrealtime() - t0 > DEADLINE)) {
                        if (lane == 0) st_agt(tmo, call);
                        bad = true;
                        break;
                    }
                    __builtin_amdgcn_s_sleep(PSLEEP);
                }
            }
            if (k < NIN) {
                if (bad) { ticket(); continue; }
                const int hb = k & 1, isw = k < 224 ? (k >> 1) & 1 : 1;
                const int t = k < 224 ? (k >> 5) + 1 : (k - 224) >> 4, lb = k < 224 ? (k >> 2) & 7 : 8 + ((k >> 1) & 7);
                const unsigned ne = min(meta & 511u, 256u), base = ((meta >> 16) & 31u) * 0x04000400u;
                const bool raw = (meta >> 24) & 1;
                const char* src = p.ws[s] + OFF_P + (size_t)((t * 64 + (isw ? 32 + (rank & 1) * 16 : (rank >> 1) * 8) + lb) * 2 + hb) * 8192;
                pull(src, raw ? 512 : 448 + ((ne + 7) >> 3), 2, true);
                ticket();
                u32x4 h[8];
                if (!raw) {
                    const u64 ex = *(const u64*)(reg + 7168 + lane * 8);
                    u32x4 top = {0, 0, 0, 0};
#pragma unroll
                    for (int c = 0; c < 7; ++c) h[c] = *(const u32x4*)(reg + c * 1024 + lane * 16);
                    cfence();
#pragma unroll
                    for (int c = 0; c < 7; ++c) {
                        top |= (h[c] & 0x60006000u) >> (13 - 2 * c);
                        *(u32x4*)(reg + c * 1024 + lane * 16) = (h[c] & 0x9fff9fffu) + base;
                    }
                    *(u32x4*)(reg + 7168 + lane * 16) = ((top & 0x1fff1fffu) | ((top & 0x20002000u) << 2)) + base;
                    cfence();
#pragma unroll
                    for (int j = 0; j < 4; ++j) {
                        const unsigned en = (unsigned)(ex >> (16 * j)) & 0xffffu;
                        if (lane * 4 + j < ne)
                            __hip_atomic_fetch_sub((unsigned*)(reg + (en & 4094u) * 2), ((en >> 12) + 1) << (10 + 16 * (en & 1)),
                                                   __ATOMIC_RELAXED, __HIP_MEMORY_SCOPE_WAVEFRONT);
                    }
                }
                cfence();
                const int blk = isw * 8 + lb;
                char* dst = myws + OFF_I + (t * 8 + s) * KC_B + blk * 16384 + hb * 8192 + lane * 16;
#pragma unroll
                for (int c = 0; c < 8; ++c) h[c] = *(const u32x4*)(reg + c * 1024 + lane * 16);
#pragma unroll
                for (int c = 0; c < 8; ++c) st16_loc(dst + c * 1024, h[c]);
                vm_wait0();
                if (lane == 0) st_sys(myws + OFF_XF + blk * 512 + (t * 8 + s) * 8 + hb * 4, call);
                continue;
            }
            const int strip = (meta >> 25) * 8 + (k & 7);
            f32x4 v[8];
            if (!bad) {
                const unsigned L = min(meta & 255u, 128u), ne = min((meta >> 8) & 255u, (unsigned)EXC_MAX), elo = (meta >> 16) & 255u;
                const bool raw = (meta >> 24) & 1;
                const unsigned n16 = 16 + (raw ? 4 * L : (13 * L * 4 + ne * 2 + 15) >> 4);
                pull(p.ws[s] + OFF_O + (size_t)strip * SLOT_B, n16, 1, k < TSW);
                ticket();
                unsigned* const dv = (unsigned*)(reg + 256);
                if (!raw) {
                    const unsigned ex = dv[13 * L + lane];
                    cfence();
                    for (unsigned pp = lane; pp < L; pp += 64) {
                        unsigned t[13];
#pragma unroll
                        for (int i = 0; i < 13; ++i) t[i] = dv[i * L + pp];
#pragma unroll
                        for (int i = 0; i < 13; ++i) dv[i * L + pp] = (t[i] & 0x3ffffffu) + (elo << 23);
#pragma unroll
                        for (int k = 0; k < 3; ++k)
                            dv[(13 + k) * L + pp] = ((t[4 * k] >> 26) | ((t[4 * k + 1] >> 26) << 6) | ((t[4 * k + 2] >> 26) << 12) |
                                                     ((t[4 * k + 3] >> 26) << 18) | (((t[12] >> (26 + 2 * k)) & 3u) << 24)) + (elo << 23);
                    }
                    cfence();
                    if (lane * 2 < ne) dv[ex & 2047u] -= (((ex >> 11) & 31u) + 1) << 23;
                    if (lane * 2 + 1 < ne) dv[(ex >> 16) & 2047u] -= ((ex >> 27) + 1) << 23;
                }
                cfence();
                const unsigned bits = *(const unsigned*)(reg + lane * 4);
                unsigned off = 0;
#pragma unroll
                for (int r = 0; r < 8; ++r)
#pragma unroll
                    for (int c = 0; c < 4; ++c) {
                        const bool keep = (bits >> (r * 4 + c)) & 1;
                        const u64 m = __builtin_amdgcn_ballot_w64(keep);
                        const float val = *(const float*)(reg + 256 + (off + mbcnt64(m)) * 4);
                        v[r][c] = keep ? val : 0.f;
                        off += __builtin_popcountll(m);
                    }
            } else {
                ticket();
#pragma unroll
                for (int r = 0; r < 8; ++r) v[r] = f32x4{__builtin_nanf(""), __builtin_nanf(""), __builtin_nanf(""), __builtin_nanf("")};
            }
            const int tile = strip >> 3, rb = (tile >> 3) & 7, cb = (tile >> 6) * 8 + (tile & 7);
            float* o = p.out + (size_t)((s >> 1) * 1024 + rb * 128 + (strip & 7) * 16 + (lane >> 5) * 4) * NN
                       + (s & 1) * 2048 + cb * 128 + (lane & 31) * 4;
#pragma unroll
            for (int r = 0; r < 8; ++r) st16_wt(o + (size_t)((r & 3) + 8 * (r >> 2)) * NN, v[r]);
        }
    }
}

extern "C" void kernel_launch(int rank, int nranks, ncclComm_t comm,
                              hipStream_t stream,
                              void* const* d_in, const int* in_sizes, int n_in,
                              void* d_out, int out_size,
                              const int* in_dtypes, int out_dtype,
                              void* const* d_out_all,
                              void* const* d_ws_all, size_t ws_bytes) {
    if (nranks != NR || (unsigned)rank >= (unsigned)NR || ws_bytes < WS_NEED || n_in < 2 ||
        in_sizes[0] != MM * KS || in_sizes[1] != KS * NN || out_size != MM * NN) {
        fprintf(stderr, "kernel_launch: unexpected problem shape\n");
        return;
    }
    HIP_CHECK(hipFuncSetAttribute((const void*)gemm_ar_kernel, hipFuncAttributeMaxDynamicSharedMemorySize, LDS_BYTES));
    Params p;
    p.x = (const float*)d_in[0];
    p.w = (const float*)d_in[1];
    p.out = (float*)d_out;
    for (int i = 0; i < NR; ++i) p.ws[i] = (char*)d_ws_all[i];
    p.rank = rank;
    hipLaunchKernelGGL(gemm_ar_kernel, dim3(NWG), dim3(NTHR), LDS_BYTES, stream, p);
    HIP_CHECK(hipGetLastError());
}
```

```cpp
#include <hip/hip_runtime.h>
#include <rccl/rccl.h>
#include <cstdio>
#include <cstdint>

#define HIP_CHECK(expr) do { hipError_t _e = (expr); if (_e != hipSuccess) { \
    fprintf(stderr, "HIP error %s at %s:%d\n", hipGetErrorString(_e), __FILE__, __LINE__); \
    return; } } while (0)

typedef unsigned u32x4 __attribute__((ext_vector_type(4)));
typedef unsigned u32x32 __attribute__((ext_vector_type(32)));
typedef float f32x4 __attribute__((ext_vector_type(4)));
typedef float f32x16 __attribute__((ext_vector_type(16)));
typedef _Float16 f16x8 __attribute__((ext_vector_type(8)));
typedef unsigned long long u64;
typedef __attribute__((address_space(1))) unsigned gu32;
typedef __attribute__((address_space(1))) unsigned long long gu64;

constexpr int NR = 8;
constexpr int MM = 4096, NN = 4096, KS = 512;
constexpr int NWG = 256, NTHR = 512;
#define EW 7
#define TOK 2
#define XP (xcd)
#define G1 6
#define IW 2
#define OW 1
#define PSLEEP 8
#define STAG 8
constexpr int NSLOT = 5;
constexpr int REG_B = 9216;
constexpr int EXC_OFF = 8704, EXC_MAX = 64;
constexpr int LDS_BYTES = NSLOT * 32768;
constexpr int LDS_BAD = 8 * REG_B;
constexpr int LDS_TOK = LDS_BAD + 16;
constexpr u64 DEADLINE = 40000000ull;

constexpr size_t OFF_SEQ = 0;
constexpr size_t OFF_TMO = 32768;
constexpr size_t OFF_ORD = 32768 + 256;
constexpr size_t OFF_ETK = 32768 + 1024;
constexpr size_t OFF_XF = 65536;
constexpr int NIN = 352, NITEM = NIN + 1024;
constexpr size_t OFF_OF = 131072;
constexpr size_t OFF_DBG = 262144;
constexpr size_t OFF_I = 2u << 20;
constexpr size_t KC_B = 24 * 16384;
constexpr size_t OFF_P = OFF_I + 64 * KC_B;
constexpr size_t OFF_O = OFF_P + (8u << 20);
constexpr size_t SLOT_B = 8448;
constexpr size_t WS_NEED = OFF_O + (size_t)1024 * SLOT_B;
static_assert(OFF_OF + NR * NITEM * 8 <= OFF_DBG, "layout");

struct Params {
    const float* x;
    const float* w;
    float* out;
    char* ws[NR];
    int rank;
};

__device__ __forceinline__ void st16_sys(void* p, u32x4 v) {
    asm volatile("global_store_dwordx4 %0, %1, off sc0 sc1\n\ts_nop 1" :: "v"(p), "v"(v) : "memory");
}
#define OSC "sc1 nt"
#define USC "sc0 sc1"
#define PSC "sc0 sc1 nt"
__device__ __forceinline__ void st16_wt(void* p, f32x4 v) {
    asm volatile("global_store_dwordx4 %0, %1, off " OSC "\n\ts_nop 1" :: "v"(p), "v"(v) : "memory");
}
__device__ __forceinline__ void st16_loc(void* p, u32x4 v) {
    asm volatile("global_store_dwordx4 %0, %1, off " USC "\n\ts_nop 1" :: "v"(p), "v"(v) : "memory");
}
__device__ __forceinline__ void cfence() { asm volatile("" ::: "memory"); }
__device__ __forceinline__ void vm_wait0() { asm volatile("s_waitcnt vmcnt(0)" ::: "memory"); }
__device__ __forceinline__ u64 ld_sys64(const void* p) {
    return __hip_atomic_load((gu64*)p, __ATOMIC_RELAXED, __HIP_MEMORY_SCOPE_SYSTEM);
}
__device__ __forceinline__ void st_sys64(void* p, u64 v) {
    __hip_atomic_store((gu64*)p, v, __ATOMIC_RELAXED, __HIP_MEMORY_SCOPE_SYSTEM);
}
__device__ __forceinline__ void st_sys(void* p, unsigned v) {
    __hip_atomic_store((gu32*)p, v, __ATOMIC_RELAXED, __HIP_MEMORY_SCOPE_SYSTEM);
}
__device__ __forceinline__ unsigned ld_agt(const void* p) {
    return __hip_atomic_load((gu32*)p, __ATOMIC_RELAXED, __HIP_MEMORY_SCOPE_AGENT);
}
__device__ __forceinline__ void st_agt(void* p, unsigned v) {
    __hip_atomic_store((gu32*)p, v, __ATOMIC_RELAXED, __HIP_MEMORY_SCOPE_AGENT);
}
#define GSC "sc0 sc1"
__device__ __forceinline__ void glds4k(const char* sbase, unsigned voff, unsigned lds_dst) {
    unsigned keep;
    asm volatile("s_mov_b32 %0, m0\n\ts_mov_b32 m0, %3\n\ts_nop 3\n\t"
                 "global_load_lds_dwordx4 %1, %2 " GSC "\n\t"
                 "global_load_lds_dwordx4 %1, %2 offset:1024 " GSC "\n\t"
                 "global_load_lds_dwordx4 %1, %2 offset:2048 " GSC "\n\t"
                 "global_load_lds_dwordx4 %1, %2 offset:3072 " GSC "\n\t"
                 "s_mov_b32 m0, %0"
                 : "=&s"(keep) : "v"(voff), "s"(sbase), "s"(lds_dst) : "memory");
}
__device__ __forceinline__ void glds1k(const char* sbase, unsigned voff, unsigned lds_dst) {
    unsigned keep;
    asm volatile("s_mov_b32 %0, m0\n\ts_mov_b32 m0, %3\n\ts_nop 3\n\t"
                 "global_load_lds_dwordx4 %1, %2 " PSC "\n\t"
                 "s_mov_b32 m0, %0"
                 : "=&s"(keep) : "v"(voff), "s"(sbase), "s"(lds_dst) : "memory");
}
__device__ __forceinline__ unsigned mbcnt64(u64 m) {
    return __builtin_amdgcn_mbcnt_hi((unsigned)(m >> 32), __builtin_amdgcn_mbcnt_lo((unsigned)m, 0u));
}
__device__ __forceinline__ const char* uni_ptr(const char* p) {
    u64 v = (u64)p;
    unsigned lo = __builtin_amdgcn_readfirstlane((unsigned)v), hi = __builtin_amdgcn_readfirstlane((unsigned)(v >> 32));
    return (const char*)(((u64)hi << 32) | lo);
}
__device__ __forceinline__ u32x4 cvt8(f32x4 a, f32x4 b) {
    f16x8 h;
    h[0] = (_Float16)a[0]; h[1] = (_Float16)a[1]; h[2] = (_Float16)a[2]; h[3] = (_Float16)a[3];
    h[4] = (_Float16)b[0]; h[5] = (_Float16)b[1]; h[6] = (_Float16)b[2]; h[7] = (_Float16)b[3];
    return __builtin_bit_cast(u32x4, h);
}

typedef unsigned short u16x2 __attribute__((ext_vector_type(2)));
__device__ __forceinline__ unsigned pk_subsat(unsigned a, unsigned b) {
    return __builtin_bit_cast(unsigned, __builtin_elementwise_sub_sat(__builtin_bit_cast(u16x2, a), __builtin_bit_cast(u16x2, b)));
}
__device__ __forceinline__ unsigned pk_max(unsigned a, unsigned b) {
    return __builtin_bit_cast(unsigned, __builtin_elementwise_max(__builtin_bit_cast(u16x2, a), __builtin_bit_cast(u16x2, b)));
}
__device__ __forceinline__ unsigned pk_min(unsigned a, unsigned b) {
    return __builtin_bit_cast(unsigned, __builtin_elementwise_min(__builtin_bit_cast(u16x2, a), __builtin_bit_cast(u16x2, b)));
}

__device__ __forceinline__ unsigned pk_sub(unsigned a, unsigned b) {
    return __builtin_bit_cast(unsigned, (u16x2)(__builtin_bit_cast(u16x2, a) - __builtin_bit_cast(u16x2, b)));
}
__device__ __forceinline__ unsigned ubfe(unsigned v, unsigned o, unsigned n) { return __builtin_amdgcn_ubfe(v, o, n); }
template <int BITS> __device__ __forceinline__ unsigned wave_scan(unsigned v, unsigned& tot) {
    unsigned pre = 0;
    tot = 0;
#pragma unroll
    for (int b = 0; b < BITS; ++b) {
        const u64 m = __builtin_amdgcn_ballot_w64((v >> b) & 1);
        pre += mbcnt64(m) << b;
        tot += __builtin_popcountll(m) << b;
    }
    return pre;
}
__device__ __forceinline__ void put_bits(char* base, unsigned off, unsigned bits) {
    const u64 v = (u64)bits << (off & 31);
    unsigned* w = (unsigned*)base + (off >> 5);
    __hip_atomic_fetch_or(w, (unsigned)v, __ATOMIC_RELAXED, __HIP_MEMORY_SCOPE_WAVEFRONT);
    __hip_atomic_fetch_or(w + 1, (unsigned)(v >> 32), __ATOMIC_RELAXED, __HIP_MEMORY_SCOPE_WAVEFRONT);
}
__device__ __forceinline__ unsigned get_bits(const char* base, unsigned off) {
    const unsigned* w = (const unsigned*)base + (off >> 5);
    return __builtin_amdgcn_alignbit(w[1], w[0], off & 31);
}

#define STAMP(i) do {} while (0)

struct Blk { int isw, grp, lb, t; };
__device__ __forceinline__ Blk decode_blk(int v) {
    Blk b;
    if (v < 384) {
        b.t = v / 48; int e = v % 48, e3 = e / 3, r3 = e % 3;
        if (r3 == 2) { b.isw = 1; b.grp = e3 & 1; b.lb = e3 >> 1; }
        else { int idx = e3 * 2 + r3; b.isw = 0; b.grp = idx & 3; b.lb = idx >> 2; }
    } else {
        int u = v - 384; b.t = u >> 4; int e = u & 15;
        b.isw = 1; b.grp = e & 1; b.lb = 8 + (e >> 1);
    }
    return b;
}

__global__ void __launch_bounds__(NTHR, 2) gemm_ar_kernel(Params p) {
    extern __shared__ __attribute__((aligned(16))) char lds[];
    const int tid = threadIdx.x, lane = tid & 63;
    const int wv = __builtin_amdgcn_readfirstlane(tid >> 6);
    const int rank = p.rank;
    char* const myws = p.ws[rank];
    const int xcd = blockIdx.x & 7, u = blockIdx.x >> 3;
    const unsigned lds0 = (unsigned)(uintptr_t)(__attribute__((address_space(3))) char*)lds;
    char* const tmo = myws + OFF_TMO;
    STAMP(0);

    if (tid < 2) ((unsigned*)(lds + LDS_TOK))[tid] = 0;
    __syncthreads();
    char* const seqp = myws + OFF_SEQ + (blockIdx.x * 8 + wv) * 16;
    unsigned call = ld_agt(seqp);
    auto next_call = [&]() {
        call = __builtin_amdgcn_readfirstlane(call) + 1;
        if (lane == 0) st_agt(seqp, call);
    };

    if (u < 16) {
        next_call();
        const int stage = u >> 3, ul = u & 7;
        const int rb = (xcd >> 1) * 2 + (ul >> 2);
        const int cb = stage * 8 + (xcd & 1) * 4 + (ul & 3);
        const int wm = wv & 3, kh = wv >> 2;
        const char* const gsrc = myws + OFF_I + (size_t)(wv < 4 ? rb : 8 + cb) * 16384;
        const unsigned voff = (wv & 3) * 4096 + lane * 16;
        const int fpos = (lane & ~7) | ((rank + lane) & 7);
        const char* const xf = myws + OFF_XF + rb * 512 + fpos * 8;
        const char* const wf = myws + OFF_XF + (8 + cb) * 512 + fpos * 8;
        u64 ready = 0;
        unsigned timed_out = 0;
        auto ensure = [&](int n) {
            if ((ready >> n) & 1) return;
            const u64 t0 = __builtin_amdgcn_s_memrealtime();
            for (unsigned spin = 1;; ++spin) {
                const u64 a = ld_sys64(xf), b = ld_sys64(wf);
                ready = __builtin_amdgcn_ballot_w64(min(min((unsigned)a, (unsigned)(a >> 32)), min((unsigned)b, (unsigned)(b >> 32))) >= call);
                if ((ready >> n) & 1) break;
                if ((spin & 255) == 0 && (__builtin_amdgcn_readfirstlane(ld_agt(tmo)) == call ||
                                          __builtin_amdgcn_s_memrealtime() - t0 > DEADLINE)) {
                    if (lane == 0) st_agt(tmo, call);
                    vm_wait0();
                    ready = ~0ull;
                    timed_out = 1;
                    break;
                }
                __builtin_amdgcn_s_sleep(8);
            }
        };
        auto issue = [&](int n, int slot) {
            glds4k(gsrc + (size_t)((n & ~7) | ((rank + n) & 7)) * KC_B, voff, lds0 + slot * 32768 + wv * 4096);
        };
        struct Frag { f16x8 a[2], b[2][4]; };
        auto rd = [&](Frag& f, int slot) {
            const char* sa = lds + slot * 32768 + lane * 16;
#pragma unroll
            for (int qq = 0; qq < 2; ++qq) {
                f.a[qq] = *(const f16x8*)(sa + (wm * 4 + kh * 2 + qq) * 1024);
#pragma unroll
                for (int c = 0; c < 4; ++c) f.b[qq][c] = *(const f16x8*)(sa + 16384 + ((kh * 2 + qq) * 4 + c) * 1024);
            }
        };
        f32x16 acc[4];
#pragma unroll
        for (int c = 0; c < 4; ++c)
#pragma unroll
            for (int r = 0; r < 16; ++r) acc[c][r] = 0.f;

        __builtin_amdgcn_s_waitcnt(0);
        if (wv == 0) { for (int n = 0; n < NSLOT; ++n) ensure(n); }
        __builtin_amdgcn_s_barrier();
        asm volatile("" ::: "memory");
        STAMP(1);
        for (int n = 0; n < NSLOT; ++n) issue(n, n);
        asm volatile("s_waitcnt vmcnt(%0)" :: "i"(4 * (NSLOT - 1)) : "memory");
        __builtin_amdgcn_s_barrier();
        asm volatile("" ::: "memory");
        Frag f0, f1;
        rd(f0, 0);
        int slot = 0;
        auto step = [&](int n, Frag& cur, Frag& nxt) {
            if (wv == 0 && n + NSLOT < 64) ensure(n + NSLOT);
            if (n + NSLOT <= 64) asm volatile("s_waitcnt vmcnt(%0)" :: "i"(4 * (NSLOT - 2)) : "memory");
            else vm_wait0();
            __builtin_amdgcn_s_waitcnt(0xC07F);
            __builtin_amdgcn_s_barrier();
            asm volatile("" ::: "memory");
            const int ns = slot == NSLOT - 1 ? 0 : slot + 1;
            rd(nxt, ns);
            if (n + NSLOT < 64) issue(n + NSLOT, slot);
            slot = ns;
            __builtin_amdgcn_sched_barrier(0);
#pragma unroll
            for (int qq = 0; qq < 2; ++qq)
#pragma unroll
                for (int c = 0; c < 4; ++c)
                    acc[c] = __builtin_amdgcn_mfma_f32_32x32x16_f16(cur.a[qq], cur.b[qq][c], acc[c], 0, 0, 0);
            __builtin_amdgcn_sched_barrier(0);
        };
        for (int n = 0; n < 60; n += 2) { step(n, f0, f1); step(n + 1, f1, f0); }
        unsigned ord = 0;
        if (tid == 0) ord = __hip_atomic_fetch_add((gu32*)(myws + OFF_ORD), 1u, __ATOMIC_RELAXED, __HIP_MEMORY_SCOPE_AGENT);
        for (int n = 60; n < 64; n += 2) { step(n, f0, f1); step(n + 1, f1, f0); }
        STAMP(2);
        __syncthreads();
        f32x4 v[8];
        {
            char* dst = lds + (wv ^ 4) * REG_B + lane * 16;
            if (kh == 0) {
#pragma unroll
                for (int r = 0; r < 8; ++r) {
                    *(f32x4*)(dst + r * 1024) = f32x4{acc[0][8 + r], acc[1][8 + r], acc[2][8 + r], acc[3][8 + r]};
                    v[r] = f32x4{acc[0][r], acc[1][r], acc[2][r], acc[3][r]};
                }
            } else {
#pragma unroll
                for (int r = 0; r < 8; ++r) {
                    *(f32x4*)(dst + r * 1024) = f32x4{acc[0][r], acc[1][r], acc[2][r], acc[3][r]};
                    v[r] = f32x4{acc[0][8 + r], acc[1][8 + r], acc[2][8 + r], acc[3][8 + r]};
                }
            }
        }
        if (tid == 0) *(unsigned*)(lds + LDS_BAD) = (ord & 127) | (timed_out << 7);
        __syncthreads();
        ord = __builtin_amdgcn_readfirstlane(*(const unsigned*)(lds + LDS_BAD));
        const bool bad = ord >> 7;
        ord &= 127;
        char* const reg = lds + wv * REG_B;
#pragma unroll
        for (int r = 0; r < 8; ++r) v[r] += *(const f32x4*)(reg + r * 1024 + lane * 16);
        if (bad) {
#pragma unroll
            for (int r = 0; r < 8; ++r) v[r] = f32x4{__builtin_nanf(""), __builtin_nanf(""), __builtin_nanf(""), __builtin_nanf("")};
        }
        asm volatile("s_waitcnt lgkmcnt(0)" ::: "memory");
        u32x32 x;
        unsigned se = 0, nz = 0;
#pragma unroll
        for (int j = 31; j >= 0; --j) {
            const float val = v[j >> 2][j & 3];
            x[j] = val <= 0.f ? 0u : __float_as_uint(val);
            se += x[j] >> 23;
            nz = (nz << 1) | min(x[j], 1u);
        }
        se += __builtin_popcount(nz) << 20;
#pragma unroll
        for (int o = 32; o > 0; o >>= 1) se += (unsigned)__shfl_xor((int)se, o);
        const unsigned elo = __builtin_amdgcn_readfirstlane(
            min(max((int)((float)(se & 0xfffffu) * __builtin_amdgcn_rcpf((float)max(se >> 20, 1u))) - 6, 1), 245));
        unsigned pl[32], sb[4], sl[4], bits = 0, hb = 0, ne = 0;
#pragma unroll
        for (int g = 3; g >= 0; --g) {
            unsigned acc = 0, ls = 0;
#pragma unroll
            for (int j = g * 8 + 7; j >= g * 8; --j) {
                const unsigned z = min((x[j] >> 23) - elo, 11u), in = ubfe(0x7ffu, z, 1), esc = ubfe(0x63fu, z, 1), ln = __umul24(esc, 3);
                pl[j] = (x[j] & ~0x800000u) | ((0xfd6000u << z) & 0x800000u);
                bits = (bits << 1) | in;
                hb = (hb << in) | ubfe(0x73fu, z, 1);
                acc = (acc << ln) | ubfe(0x4003eb1au, __umul24(z, 3), ln);
                ls += esc;
            }
            sb[g] = acc;
            sl[g] = ls;
        }
        nz &= ~bits;
        for (u64 bal = __builtin_amdgcn_ballot_w64(nz != 0); bal && ne <= EXC_MAX; bal &= bal - 1) {
            const unsigned l = __builtin_ctzll(bal);
            for (unsigned m = __builtin_amdgcn_readlane(nz, l); m; m &= m - 1, ++ne) {
                const unsigned j = __builtin_ctz(m);
                if (lane == l) *(u64*)(reg + 256 + ne * 8) = ((u64)x[j] << 32) | (l * 32 + j);
            }
        }
#pragma unroll
        for (int j = 0; j < 32; ++j) v[j >> 2][j & 3] = __uint_as_float(x[j]);
        unsigned off, nesc;
        const unsigned pre = wave_scan<6>(__builtin_popcount(bits), off);
        unsigned ep = wave_scan<6>(sl[0] + sl[1] + sl[2] + sl[3], nesc);
        const unsigned a16 = 256 + ne * 8, bs = (a16 + 3 * off + 3) & ~3u;
        unsigned n16 = (bs * 8 + off + 3 * nesc + 127) >> 7;
        const int tile = stage * 64 + rb * 8 + (cb & 7), ts = wm * 2 + kh;
        char* const dst = myws + OFF_O + (size_t)(tile * 8 + ts) * SLOT_B;
        if (ne > EXC_MAX || n16 >= 512) {
            ne = n16 = 0;
#pragma unroll
            for (int r = 0; r < 8; ++r) st16_sys(dst + r * 1024 + lane * 16, __builtin_bit_cast(u32x4, v[r]));
        } else {
            *(unsigned*)(reg + lane * 4) = bits;
#pragma unroll
            for (int i = 0; i < 4; ++i) *(unsigned*)(reg + bs + i * 256 + lane * 4) = 0;
            unsigned pos = pre;
#pragma unroll
            for (int j = 0; j < 32; ++j) {
                const unsigned t = __builtin_amdgcn_sbfe(bits, j, 1), at = (pos & t) | (4200u & ~t);
                *(unsigned short*)(reg + a16 + at * 2) = (unsigned short)pl[j];
                *(unsigned char*)(reg + a16 + off * 2 + at) = (unsigned char)(pl[j] >> 16);
                pos -= t;
            }
            cfence();
            put_bits(reg + bs, pre, hb);
#pragma unroll
            for (int g = 0; g < 4; ++g) {
                put_bits(reg + bs, off + 3 * ep, sb[g]);
                ep += sl[g];
            }
            cfence();
            for (unsigned i = lane; i < n16; i += 64) st16_sys(dst + i * 16, *(const u32x4*)(reg + i * 16));
        }
        const u64 flagv = ((u64)call << 32) | n16 | (ne << 9) | (elo << 16) | ((unsigned)tile << 25);
        STAMP(3);
        vm_wait0();
        if (lane < NR && lane != rank) st_sys64(p.ws[lane] + OFF_OF + (rank * NITEM + NIN + ord * 8 + ts) * 8, flagv);
        STAMP(4);
        float* o = p.out + (size_t)((rank >> 1) * 1024 + rb * 128 + wm * 32 + kh * 16 + (lane >> 5) * 4) * NN
                   + (rank & 1) * 2048 + cb * 128 + (lane & 31) * 4;
#pragma unroll
        for (int r = 0; r < 8; ++r) st16_wt(o + (size_t)((r & 3) + 8 * (r >> 2)) * NN, v[r]);
        STAMP(5);
        return;
    }

    char* const reg = lds + wv * REG_B;
    {
        const int o = wv * 128 + (u - 16) * 8 + xcd, hb = o & 1;
        const Blk b = decode_blk(o >> 1);
        for (int i = 0; i < wv; ++i) __builtin_amdgcn_s_sleep(STAG);
        u32x4 h[8];
        {
            f32x4 f[2][8];
            if (b.isw) {
                const float* src = p.w + (size_t)(b.t * 64 + (lane >> 5) * 32 + hb * 16) * NN + b.grp * 2048 + b.lb * 128 + (lane & 31) * 4;
#pragma unroll
                for (int j = 0; j < 16; ++j) f[j >> 3][j & 7] = *(const f32x4*)(src + (size_t)j * NN);
#pragma unroll
                for (int g = 0; g < 2; ++g)
#pragma unroll
                    for (int c = 0; c < 4; ++c)
                        h[g * 4 + c] = cvt8(f32x4{f[g][0][c], f[g][1][c], f[g][2][c], f[g][3][c]},
                                            f32x4{f[g][4][c], f[g][5][c], f[g][6][c], f[g][7][c]});
            } else {
                const float* src = p.x + (size_t)(b.grp * 1024 + b.lb * 128 + hb * 64 + (lane & 31)) * KS + b.t * 64 + (lane >> 5) * 32;
#pragma unroll
                for (int j = 0; j < 16; ++j) f[j >> 3][j & 7] = *(const f32x4*)(src + (size_t)(j >> 3) * 32 * KS + (j & 7) * 4);
#pragma unroll
                for (int g = 0; g < 2; ++g)
#pragma unroll
                    for (int q = 0; q < 4; ++q) h[g * 4 + q] = cvt8(f[g][2 * q], f[g][2 * q + 1]);
            }
        }
        next_call();
        STAMP(15);
        const int nd = b.isw ? 4 : 2, blk = b.isw * 8 + b.lb;
        const int dl = (b.isw ? b.grp + 2 * lane : b.grp * 2 + lane) & 7;
        const size_t doff = OFF_I + (b.t * 8 + rank) * KC_B + blk * 16384 + hb * 8192 + lane * 16;
        const size_t foff = OFF_XF + blk * 512 + (b.t * 8 + rank) * 8 + hb * 4;
        if (o < 96) {
            for (int i = 0; i < nd; ++i) {
                const int ii = (i + b.lb + hb) & (nd - 1);
                char* dst = p.ws[(b.isw ? b.grp + 2 * ii : b.grp * 2 + ii) & 7] + doff;
#pragma unroll
                for (int c = 0; c < 8; ++c) st16_sys(dst + c * 1024, h[c]);
            }
            vm_wait0();
            if (lane < nd) st_sys(p.ws[dl] + foff, call);
        } else {
            if (b.grp == (b.isw ? rank & 1 : rank >> 1)) {
#pragma unroll
                for (int c = 0; c < 8; ++c) st16_loc(myws + doff + c * 1024, h[c]);
            }
            unsigned mx = 0, se = 0;
#pragma unroll
            for (int c = 0; c < 8; ++c) {
                *(u32x4*)(reg + c * 1024 + lane * 16) = h[c];
#pragma unroll
                for (int j = 0; j < 4; ++j) {
                    const unsigned e = (h[c][j] >> 10) & 0x001f001fu;
                    mx = pk_max(mx, e);
                    se += e;
                }
            }
            mx = max(mx >> 16, mx & 0xffffu);
#pragma unroll
            for (int d = 32; d > 0; d >>= 1) mx = max(mx, (unsigned)__shfl_xor((int)mx, d));
            wave_scan<11>((se >> 16) + (se & 0xffffu), se);
            const unsigned lo = max(max((int)__builtin_amdgcn_readfirstlane(mx) - 10, (int)(se >> 12) - 6), 0), base = lo * 0x04000400u;
            unsigned q[32], sb[8], sl[8], cnt = 0;
            unsigned em[2] = {0, 0};
#pragma unroll
            for (int c = 0; c < 8; ++c) {
                unsigned acc = 0, ls = 0;
#pragma unroll
                for (int j = 3; j >= 0; --j) {
                    const unsigned wd = h[c][j], ea = wd & 0x7c007c00u, zq = pk_subsat(ea, base);
                    const unsigned c2 = pk_min(pk_sub(zq, 0x18001800u), 0x0c000c00u), l = (c2 & (c2 >> 1)) >> 10;
                    const unsigned c3 = ((zq >> 10) - 3 * ((zq >> 13) & 0x00010001u)) & (l * 7), ln = l * 3;
                    q[c * 4 + j] = (wd & 0x83ff83ffu) | c2;
                    em[c >> 2] |= pk_min(pk_subsat(base, ea), 0x00010001u) << ((c & 3) * 4 + j);
                    acc = (acc << (ln >> 16)) | (c3 >> 16);
                    acc = (acc << (ln & 31)) | (c3 & 7);
                    ls += l;
                }
                sb[c] = acc;
                sl[c] = cnt;
                cnt += (ls >> 16) + (ls & 0xffffu);
            }
            unsigned nesc;
            const unsigned pre = wave_scan<7>(cnt, nesc);
            u64 m = ((u64)em[1] << 32) | em[0];
            cfence();
            unsigned ne = 0;
            bool raw = false;
            for (u64 bal; (bal = __builtin_amdgcn_ballot_w64(m != 0)) != 0 && !raw;) {
                bool far = false;
                if (m != 0) {
                    const unsigned bt = __builtin_ctzll(m), wi = (bt & 15) + (bt >> 5) * 16;
                    m &= m - 1;
                    const unsigned idx = (wi >> 2) * 512 + lane * 8 + (wi & 3) * 2 + ((bt >> 4) & 1);
                    const unsigned dlt = lo - 1 - ((*(const unsigned short*)(reg + idx * 2) >> 10) & 31u), pos = ne + mbcnt64(bal);
                    far = dlt > 15;
                    if (pos < 256) *(unsigned short*)(reg + EXC_OFF + pos * 2) = (unsigned short)(idx | (dlt << 12));
                }
                ne += __builtin_popcountll(bal);
                raw = ne > 256 || __builtin_amdgcn_ballot_w64(far) != 0;
            }
            cfence();
            char* dst = myws + OFF_P + (size_t)((b.t * 64 + (b.isw ? 32 + b.grp * 16 : b.grp * 8) + b.lb) * 2 + hb) * 8192 + lane * 16;
            unsigned ext = (16 * ne + 3 * nesc + 127) >> 7;
            if (raw || ext > 96) {
                raw = true;
                ne = ext = 0;
#pragma unroll
                for (int c = 0; c < 8; ++c) st16_sys(dst + c * 1024, h[c]);
            } else {
#pragma unroll
                for (int g = 0; g < 2; ++g) {
#pragma unroll
                    for (int i = 0; i < 12; ++i) q[g * 13 + i] |= (q[26 + g * 3 + (i >> 2)] << (12 - 3 * (i & 3))) & 0x70007000u;
#pragma unroll
                    for (int i = 0; i < 3; ++i) q[g * 13 + 12] |= (q[26 + g * 3 + i] & 0x80008000u) >> (3 - i);
                }
#pragma unroll
                for (int c = 0; c < 6; ++c) st16_sys(dst + c * 1024, u32x4{q[c * 4], q[c * 4 + 1], q[c * 4 + 2], q[c * 4 + 3]});
                u64 ex = *(const u64*)(reg + EXC_OFF + lane * 8);
                const int nx = (int)ne - lane * 4;
                ex = nx >= 4 ? ex : nx <= 0 ? 0 : ex & ((1ull << (16 * nx)) - 1);
                *(u64*)(reg + lane * 8) = ((u64)q[25] << 32) | q[24];
                *(u64*)(reg + 512 + lane * 8) = ex;
                *(u32x4*)(reg + 1024 + lane * 16) = u32x4{0, 0, 0, 0};
                cfence();
#pragma unroll
                for (int c = 0; c < 8; ++c) put_bits(reg + 512, 16 * ne + 3 * (pre + sl[c]), sb[c]);
                cfence();
                if (lane < 32 + ext) st16_sys(dst + 6144, *(const u32x4*)(reg + lane * 16));
                if (lane + 32 < ext) st16_sys(dst + 7168, *(const u32x4*)(reg + 1024 + lane * 16));
            }
            vm_wait0();
            const int item = hb + 2 * (b.lb >= 8 ? 112 + b.t * 8 + b.lb - 8
                                       : b.lb < G1 ? (b.t - 1) * (8 + G1) + b.lb * 2 + b.isw
                                       : b.isw ? (b.t - 1) * (8 + G1) + G1 + b.lb : 7 * (8 + G1) + (b.t - 1) * (8 - G1) + b.lb - G1);
            if (lane < nd) {
                if (dl == rank) st_sys(myws + foff, call);
                else st_sys64(p.ws[dl] + OFF_OF + (rank * NITEM + item) * 8, ((u64)call << 32) | ne | (ext << 9) | (lo << 16) | ((unsigned)raw << 24));
            }
        }
        STAMP(1);
    }

    {
        __builtin_amdgcn_s_waitcnt(0);
        const int s = (XP) & 7;
        if (s == rank || wv >= EW) return;
        unsigned* const tok = (unsigned*)(lds + LDS_TOK);
        bool bad = false;
        unsigned tk = 0;
        auto ticket = [&]() {
            if (lane == 0) tk = __hip_atomic_fetch_add((gu32*)(myws + OFF_ETK + xcd * 128), 1u, __ATOMIC_RELAXED, __HIP_MEMORY_SCOPE_AGENT);
        };
        auto pull = [&](const char* src, unsigned n16, unsigned wgt) {
            unsigned my = 0;
            if (lane == 0) my = __hip_atomic_fetch_add(tok, wgt, __ATOMIC_RELAXED, __HIP_MEMORY_SCOPE_WORKGROUP);
            my = __builtin_amdgcn_readfirstlane(my) + wgt;
            while ((int)(my - __builtin_amdgcn_readfirstlane(__hip_atomic_load(tok + 1, __ATOMIC_RELAXED, __HIP_MEMORY_SCOPE_WORKGROUP))) > TOK)
                __builtin_amdgcn_s_sleep(1);
            src = uni_ptr(src);
            for (unsigned it = 0; it * 64 < n16; ++it)
                if (it * 64 + lane < n16) glds1k(src + it * 1024, lane * 16, lds0 + wv * REG_B + it * 1024);
            vm_wait0();
            if (lane == 0) __hip_atomic_fetch_add(tok + 1, wgt, __ATOMIC_RELAXED, __HIP_MEMORY_SCOPE_WORKGROUP);
        };
        ticket();
        for (int n = 0;; ++n) {
            const unsigned k = __builtin_amdgcn_readfirstlane(tk) - (call - 1) * (NITEM + 16 * EW);
            if (k >= NITEM) break;
            const char* const fp = myws + OFF_OF + (s * NITEM + k) * 8;
            unsigned meta = ((k - NIN) >> 3) << 25;
            if (!bad) {
                const u64 t0 = __builtin_amdgcn_s_memrealtime();
                for (unsigned spin = 1;; ++spin) {
                    const u64 fl = ld_sys64(fp);
                    if (__builtin_amdgcn_readfirstlane((unsigned)(fl >> 32)) >= call) { meta = __builtin_amdgcn_readfirstlane((unsigned)fl); break; }
                    if ((spin & 255) == 0 && (__builtin_amdgcn_readfirstlane(ld_agt(tmo)) == call ||
                                              __builtin_amdgcn_s_memrealtime() - t0 > DEADLINE)) {
                        if (lane == 0) st_agt(tmo, call);
                        bad = true;
                        break;
                    }
                    __builtin_amdgcn_s_sleep(PSLEEP);
                }
            }
            if (k < NIN) {
                if (bad) { ticket(); continue; }
                const int hb = k & 1, kb = k >> 1;
                int isw = 1, t = (kb - 112) >> 3, lb = 8 + (kb & 7);
                if (kb < 7 * (8 + G1)) {
                    const int e = kb % (8 + G1);
                    t = kb / (8 + G1) + 1;
                    isw = e < 2 * G1 ? e & 1 : 1;
                    lb = e < 2 * G1 ? e >> 1 : e - G1;
                } else if (kb < 112) {
                    t = (kb - 7 * (8 + G1)) / (8 - G1) + 1;
                    isw = 0;
                    lb = G1 + (kb - 7 * (8 + G1)) % (8 - G1);
                }
                const unsigned ne = min(meta & 511u, 256u), base = ((meta >> 16) & 31u) * 0x04000400u;
                const bool raw = (meta >> 24) & 1;
                const char* src = p.ws[s] + OFF_P + (size_t)((t * 64 + (isw ? 32 + (rank & 1) * 16 : (rank >> 1) * 8) + lb) * 2 + hb) * 8192;
                pull(src, raw ? 512 : 416 + min((meta >> 9) & 127u, 96u), IW);
                ticket();
                u32x4 h[8];
                if (!raw) {
                    const u64 ex = *(const u64*)(reg + 6656 + lane * 8), q2 = *(const u64*)(reg + 6144 + lane * 8);
                    unsigned q[32], sl[8], cnt = 0, tot;
#pragma unroll
                    for (int c = 0; c < 6; ++c) {
                        const u32x4 t4 = *(const u32x4*)(reg + c * 1024 + lane * 16);
#pragma unroll
                        for (int j = 0; j < 4; ++j) q[c * 4 + j] = t4[j];
                    }
                    q[24] = (unsigned)q2;
                    q[25] = (unsigned)(q2 >> 32);
#pragma unroll
                    for (int i = 0; i < 6; ++i) {
                        const unsigned* cr = q + (i / 3) * 13;
                        q[26 + i] = (cr[12] << (3 - i % 3)) & 0x80008000u;
#pragma unroll
                        for (int j = 0; j < 4; ++j) q[26 + i] |= (cr[(i % 3) * 4 + j] & 0x70007000u) >> (12 - 3 * j);
                    }
#pragma unroll
                    for (int c = 0; c < 8; ++c) {
                        sl[c] = cnt;
#pragma unroll
                        for (int j = 0; j < 4; ++j) cnt += __builtin_popcount(q[c * 4 + j] & (q[c * 4 + j] >> 1) & 0x04000400u);
                    }
                    const unsigned pre = wave_scan<7>(cnt, tot);
#pragma unroll
                    for (int c = 0; c < 8; ++c) {
                        unsigned acc = get_bits(reg + 6656, 16 * ne + 3 * (pre + sl[c]));
#pragma unroll
                        for (int j = 0; j < 4; ++j) {
                            const unsigned wd = q[c * 4 + j], c2 = (wd >> 10) & 0x00030003u, l = c2 & (c2 >> 1), ln = l * 3, mk = l * 15;
                            unsigned c3 = acc & 7;
                            acc >>= ln & 31;
                            c3 |= (acc & 7) << 16;
                            acc >>= ln >> 16;
                            c3 += 3 * ((c3 >> 1) & (c3 >> 2) & 0x00010001u);
                            h[c][j] = (wd & 0x83ff83ffu) + (((c3 & mk) | ((c2 + 0x00060006u) & ~mk)) << 10) + base;
                        }
                    }
                    cfence();
#pragma unroll
                    for (int c = 0; c < 8; ++c) *(u32x4*)(reg + c * 1024 + lane * 16) = h[c];
                    cfence();
#pragma unroll
                    for (int j = 0; j < 4; ++j) {
                        const unsigned en = (unsigned)(ex >> (16 * j)) & 0xffffu;
                        if (lane * 4 + j < ne)
                            __hip_atomic_fetch_sub((unsigned*)(reg + (en & 4094u) * 2), ((en >> 12) + 1) << (10 + 16 * (en & 1)),
                                                   __ATOMIC_RELAXED, __HIP_MEMORY_SCOPE_WAVEFRONT);
                    }
                }
                cfence();
                const int blk = isw * 8 + lb;
                char* dst = myws + OFF_I + (t * 8 + s) * KC_B + blk * 16384 + hb * 8192 + lane * 16;
#pragma unroll
                for (int c = 0; c < 8; ++c) h[c] = *(const u32x4*)(reg + c * 1024 + lane * 16);
#pragma unroll
                for (int c = 0; c < 8; ++c) st16_loc(dst + c * 1024, h[c]);
                vm_wait0();
                if (lane == 0) st_sys(myws + OFF_XF + blk * 512 + (t * 8 + s) * 8 + hb * 4, call);
                continue;
            }
            const int strip = (meta >> 25) * 8 + (k & 7);
            f32x4 v[8];
            if (!bad) {
                const unsigned n16 = meta & 511u, ne = min((meta >> 9) & 127u, (unsigned)EXC_MAX), elo = (meta >> 16) & 255u;
                pull(p.ws[s] + OFF_O + (size_t)strip * SLOT_B, n16 ? n16 : 512, OW);
                ticket();
                if (n16) {
                    const unsigned bits = *(const unsigned*)(reg + lane * 4);
                    const u64 ex = *(const u64*)(reg + 256 + lane * 8);
                    unsigned off, nesc, sl[4], es[32];
                    unsigned pos = wave_scan<6>(__builtin_popcount(bits), off);
                    const unsigned a16 = 256 + ne * 8, bs = (a16 + 3 * off + 3) & ~3u;
                    unsigned hb = get_bits(reg + bs, pos);
                    u32x32 x;
#pragma unroll
                    for (int j = 0; j < 32; ++j) {
                        x[j] = *(const unsigned short*)(reg + a16 + pos * 2) | (*(const unsigned char*)(reg + a16 + off * 2 + pos) << 16);
                        pos += (bits >> j) & 1;
                    }
#pragma unroll
                    for (int j = 0; j < 32; ++j) {
                        const unsigned in = ubfe(bits, j, 1), h = hb & in;
                        es[j] = h & (x[j] >> 23);
                        x[j] |= h << 24;
                        hb >>= in;
                        sl[j >> 3] = ((j & 7) ? sl[j >> 3] : 0u) + es[j];
                    }
                    unsigned ep = wave_scan<6>(sl[0] + sl[1] + sl[2] + sl[3], nesc);
#pragma unroll
                    for (int g = 0; g < 4; ++g) {
                        unsigned acc = get_bits(reg + bs, off + 3 * ep);
                        ep += sl[g];
#pragma unroll
                        for (int j = g * 8; j < g * 8 + 8; ++j) {
                            const unsigned ln = __umul24(es[j], 3), d = ubfe(0x56789a01u, ubfe(acc, 0, ln) << 2, 4);
                            acc >>= ln;
                            x[j] = (x[j] + ((elo + 7 - d) << 23)) & __builtin_amdgcn_sbfe(bits, j, 1);
                        }
                    }
                    for (unsigned e = 0; e < ne; ++e) {
                        const unsigned id = __builtin_amdgcn_readlane((unsigned)ex, e), val = __builtin_amdgcn_readlane((unsigned)(ex >> 32), e);
                        x[id & 31] = lane == (id >> 5) ? val : x[id & 31];
                    }
#pragma unroll
                    for (int j = 0; j < 32; ++j) v[j >> 2][j & 3] = __uint_as_float(x[j]);
                } else {
#pragma unroll
                    for (int r = 0; r < 8; ++r) v[r] = *(const f32x4*)(reg + r * 1024 + lane * 16);
                }
                cfence();
            } else {
                ticket();
#pragma unroll
                for (int r = 0; r < 8; ++r) v[r] = f32x4{__builtin_nanf(""), __builtin_nanf(""), __builtin_nanf(""), __builtin_nanf("")};
            }
            const int tile = strip >> 3, rb = (tile >> 3) & 7, cb = (tile >> 6) * 8 + (tile & 7);
            float* o = p.out + (size_t)((s >> 1) * 1024 + rb * 128 + (strip & 7) * 16 + (lane >> 5) * 4) * NN
                       + (s & 1) * 2048 + cb * 128 + (lane & 31) * 4;
#pragma unroll
            for (int r = 0; r < 8; ++r) st16_wt(o + (size_t)((r & 3) + 8 * (r >> 2)) * NN, v[r]);
        }
    }
}

extern "C" void kernel_launch(int rank, int nranks, ncclComm_t comm,
                              hipStream_t stream,
                              void* const* d_in, const int* in_sizes, int n_in,
                              void* d_out, int out_size,
                              const int* in_dtypes, int out_dtype,
                              void* const* d_out_all,
                              void* const* d_ws_all, size_t ws_bytes) {
    if (nranks != NR || (unsigned)rank >= (unsigned)NR || ws_bytes < WS_NEED || n_in < 2 ||
        in_sizes[0] != MM * KS || in_sizes[1] != KS * NN || out_size != MM * NN) {
        fprintf(stderr, "kernel_launch: unexpected problem shape\n");
        return;
    }
    HIP_CHECK(hipFuncSetAttribute((const void*)gemm_ar_kernel, hipFuncAttributeMaxDynamicSharedMemorySize, LDS_BYTES));
    Params p;
    p.x = (const float*)d_in[0];
    p.w = (const float*)d_in[1];
    p.out = (float*)d_out;
    for (int i = 0; i < NR; ++i) p.ws[i] = (char*)d_ws_all[i];
    p.rank = rank;
    hipLaunchKernelGGL(gemm_ar_kernel, dim3(NWG), dim3(NTHR), LDS_BYTES, stream, p);
    HIP_CHECK(hipGetLastError());
}
```

```cpp
#include <hip/hip_runtime.h>
#include <rccl/rccl.h>
#include <cstdio>
#include <cstdint>
#include <type_traits>

#define HIP_CHECK(expr) do { hipError_t _e = (expr); if (_e != hipSuccess) { \
    fprintf(stderr, "HIP error %s at %s:%d\n", hipGetErrorString(_e), __FILE__, __LINE__); \
    return; } } while (0)

typedef unsigned u32x4 __attribute__((ext_vector_type(4)));
typedef unsigned u32x32 __attribute__((ext_vector_type(32)));
typedef float f32x4 __attribute__((ext_vector_type(4)));
typedef float f32x16 __attribute__((ext_vector_type(16)));
typedef _Float16 f16x8 __attribute__((ext_vector_type(8)));
typedef unsigned long long u64;
typedef __attribute__((address_space(1))) unsigned gu32;
typedef __attribute__((address_space(1))) unsigned long long gu64;

constexpr int NR = 8;
constexpr int MM = 4096, NN = 4096, KS = 512;
constexpr int NWG = 256, NTHR = 512;
constexpr int EW = 7;
constexpr int TOK = 2;
constexpr int G1 = 6;
constexpr int NSLOT = 8;
constexpr int REG_B = 9216;
constexpr int EXC_OFF = 8704, EXC_MAX = 64;
constexpr int LDS_BYTES = NSLOT * 16384;
constexpr int LDS_BAD = 8 * REG_B;
constexpr int LDS_TOK = LDS_BAD + 16;
constexpr u64 DEADLINE = 40000000ull;

constexpr size_t OFF_SEQ = 0;
constexpr size_t OFF_TMO = 32768;
constexpr size_t OFF_ORD = 32768 + 256;
constexpr size_t OFF_ETK = 32768 + 1024;
constexpr size_t OFF_XF = 65536;
constexpr int NIN = 352, NITEM = NIN + 1024;
constexpr size_t OFF_OF = 131072;
constexpr size_t OFF_I = 2u << 20;
constexpr size_t KC_B = 24 * 16384;
constexpr size_t OFF_P = OFF_I + 64 * KC_B;
constexpr size_t OFF_O = OFF_P + (8u << 20);
constexpr size_t SLOT_B = 8448;
constexpr size_t WS_NEED = OFF_O + (size_t)1024 * SLOT_B;
static_assert(OFF_OF + NR * NITEM * 8 <= OFF_I, "layout");

struct Params {
    const float* x;
    const float* w;
    float* out;
    char* ws[NR];
    int rank;
};

__device__ __forceinline__ void st16_sys(void* p, u32x4 v) {
    asm volatile("global_store_dwordx4 %0, %1, off sc0 sc1\n\ts_nop 1" :: "v"(p), "v"(v) : "memory");
}
#define OSC "sc1 nt"
#define USC "sc0 sc1"
#define PSC "sc0 sc1 nt"
__device__ __forceinline__ void st16_wt(void* p, f32x4 v) {
    asm volatile("global_store_dwordx4 %0, %1, off " OSC "\n\ts_nop 1" :: "v"(p), "v"(v) : "memory");
}
__device__ __forceinline__ void st16_loc(void* p, u32x4 v) {
    asm volatile("global_store_dwordx4 %0, %1, off " USC "\n\ts_nop 1" :: "v"(p), "v"(v) : "memory");
}
__device__ __forceinline__ void cfence() { asm volatile("" ::: "memory"); }
__device__ __forceinline__ void vm_wait0() { asm volatile("s_waitcnt vmcnt(0)" ::: "memory"); }
__device__ __forceinline__ u64 ld_sys64(const void* p) {
    return __hip_atomic_load((gu64*)p, __ATOMIC_RELAXED, __HIP_MEMORY_SCOPE_SYSTEM);
}
__device__ __forceinline__ void st_sys64(void* p, u64 v) {
    __hip_atomic_store((gu64*)p, v, __ATOMIC_RELAXED, __HIP_MEMORY_SCOPE_SYSTEM);
}
__device__ __forceinline__ void st_sys(void* p, unsigned v) {
    __hip_atomic_store((gu32*)p, v, __ATOMIC_RELAXED, __HIP_MEMORY_SCOPE_SYSTEM);
}
__device__ __forceinline__ unsigned ld_agt(const void* p) {
    return __hip_atomic_load((gu32*)p, __ATOMIC_RELAXED, __HIP_MEMORY_SCOPE_AGENT);
}
__device__ __forceinline__ void st_agt(void* p, unsigned v) {
    __hip_atomic_store((gu32*)p, v, __ATOMIC_RELAXED, __HIP_MEMORY_SCOPE_AGENT);
}
#define GSC "sc0 sc1"
__device__ __forceinline__ void glds2k(const char* sbase, unsigned voff, unsigned lds_dst) {
    unsigned keep;
    asm volatile("s_mov_b32 %0, m0\n\ts_mov_b32 m0, %3\n\ts_nop 3\n\t"
                 "global_load_lds_dwordx4 %1, %2 " GSC "\n\t"
                 "global_load_lds_dwordx4 %1, %2 offset:1024 " GSC "\n\t"
                 "s_mov_b32 m0, %0"
                 : "=&s"(keep) : "v"(voff), "s"(sbase), "s"(lds_dst) : "memory");
}
__device__ __forceinline__ void glds1k(const char* sbase, unsigned voff, unsigned lds_dst) {
    unsigned keep;
    asm volatile("s_mov_b32 %0, m0\n\ts_mov_b32 m0, %3\n\ts_nop 3\n\t"
                 "global_load_lds_dwordx4 %1, %2 " PSC "\n\t"
                 "s_mov_b32 m0, %0"
                 : "=&s"(keep) : "v"(voff), "s"(sbase), "s"(lds_dst) : "memory");
}
__device__ __forceinline__ unsigned mbcnt64(u64 m) {
    return __builtin_amdgcn_mbcnt_hi((unsigned)(m >> 32), __builtin_amdgcn_mbcnt_lo((unsigned)m, 0u));
}
__device__ __forceinline__ const char* uni_ptr(const char* p) {
    u64 v = (u64)p;
    unsigned lo = __builtin_amdgcn_readfirstlane((unsigned)v), hi = __builtin_amdgcn_readfirstlane((unsigned)(v >> 32));
    return (const char*)(((u64)hi << 32) | lo);
}
__device__ __forceinline__ u32x4 cvt8(f32x4 a, f32x4 b) {
    f16x8 h;
    h[0] = (_Float16)a[0]; h[1] = (_Float16)a[1]; h[2] = (_Float16)a[2]; h[3] = (_Float16)a[3];
    h[4] = (_Float16)b[0]; h[5] = (_Float16)b[1]; h[6] = (_Float16)b[2]; h[7] = (_Float16)b[3];
    return __builtin_bit_cast(u32x4, h);
}

typedef unsigned short u16x2 __attribute__((ext_vector_type(2)));
__device__ __forceinline__ unsigned pk_subsat(unsigned a, unsigned b) {
    return __builtin_bit_cast(unsigned, __builtin_elementwise_sub_sat(__builtin_bit_cast(u16x2, a), __builtin_bit_cast(u16x2, b)));
}
__device__ __forceinline__ unsigned pk_max(unsigned a, unsigned b) {
    return __builtin_bit_cast(unsigned, __builtin_elementwise_max(__builtin_bit_cast(u16x2, a), __builtin_bit_cast(u16x2, b)));
}
__device__ __forceinline__ unsigned pk_min(unsigned a, unsigned b) {
    return __builtin_bit_cast(unsigned, __builtin_elementwise_min(__builtin_bit_cast(u16x2, a), __builtin_bit_cast(u16x2, b)));
}

__device__ __forceinline__ unsigned pk_sub(unsigned a, unsigned b) {
    return __builtin_bit_cast(unsigned, (u16x2)(__builtin_bit_cast(u16x2, a) - __builtin_bit_cast(u16x2, b)));
}
__device__ __forceinline__ unsigned ubfe(unsigned v, unsigned o, unsigned n) { return __builtin_amdgcn_ubfe(v, o, n); }
template <int BITS> __device__ __forceinline__ unsigned wave_scan(unsigned v, unsigned& tot) {
    unsigned pre = 0;
    tot = 0;
#pragma unroll
    for (int b = 0; b < BITS; ++b) {
        const u64 m = __builtin_amdgcn_ballot_w64((v >> b) & 1);
        pre += mbcnt64(m) << b;
        tot += __builtin_popcountll(m) << b;
    }
    return pre;
}
__device__ __forceinline__ void put_bits(char* base, unsigned off, unsigned bits) {
    const u64 v = (u64)bits << (off & 31);
    unsigned* w = (unsigned*)base + (off >> 5);
    __hip_atomic_fetch_or(w, (unsigned)v, __ATOMIC_RELAXED, __HIP_MEMORY_SCOPE_WAVEFRONT);
    __hip_atomic_fetch_or(w + 1, (unsigned)(v >> 32), __ATOMIC_RELAXED, __HIP_MEMORY_SCOPE_WAVEFRONT);
}
__device__ __forceinline__ unsigned get_bits(const char* base, unsigned off) {
    const unsigned* w = (const unsigned*)base + (off >> 5);
    return __builtin_amdgcn_alignbit(w[1], w[0], off & 31);
}


struct Blk { int isw, grp, lb, t; };
__device__ __forceinline__ Blk decode_blk(int v) {
    Blk b;
    if (v < 384) {
        b.t = v / 48; int e = v % 48, e3 = e / 3, r3 = e % 3;
        if (r3 == 2) { b.isw = 1; b.grp = e3 & 1; b.lb = e3 >> 1; }
        else { int idx = e3 * 2 + r3; b.isw = 0; b.grp = idx & 3; b.lb = idx >> 2; }
    } else {
        int u = v - 384; b.t = u >> 4; int e = u & 15;
        b.isw = 1; b.grp = e & 1; b.lb = 8 + (e >> 1);
    }
    return b;
}

__global__ void __launch_bounds__(NTHR, 2) gemm_ar_kernel(Params p) {
    extern __shared__ __attribute__((aligned(16))) char lds[];
    const int tid = threadIdx.x, lane = tid & 63;
    const int wv = __builtin_amdgcn_readfirstlane(tid >> 6);
    const int rank = p.rank;
    char* const myws = p.ws[rank];
    const int xcd = blockIdx.x & 7, u = blockIdx.x >> 3;
    const unsigned lds0 = (unsigned)(uintptr_t)(__attribute__((address_space(3))) char*)lds;
    char* const tmo = myws + OFF_TMO;

    if (tid < 2) ((unsigned*)(lds + LDS_TOK))[tid] = 0;
    __syncthreads();
    char* const seqp = myws + OFF_SEQ + (blockIdx.x * 8 + wv) * 16;
    unsigned call = ld_agt(seqp);
    auto next_call = [&]() {
        call = __builtin_amdgcn_readfirstlane(call) + 1;
        if (lane == 0) st_agt(seqp, call);
    };

    char* const reg = lds + wv * REG_B;
    if (u < 8 || (u >= 16 && wv >= 4)) {
        next_call();
    } else {
        const int i = wv * 64 + (u - 8) * 8 + xcd;
        const int o = u >= 16 ? 96 + wv * 128 + (u - 16) * 8 + xcd : i < 96 ? i : i + 512, hb = o & 1;
        const Blk b = decode_blk(o >> 1);
        for (int d = o < 96 ? 0 : u >= 16 ? wv : wv + 10; d > 0; --d) __builtin_amdgcn_s_sleep(32);
        u32x4 h[8];
        {
            f32x4 f[2][8];
            if (b.isw) {
                const float* src = p.w + (size_t)(b.t * 64 + (lane >> 5) * 32 + hb * 16) * NN + b.grp * 2048 + b.lb * 128 + (lane & 31) * 4;
#pragma unroll
                for (int j = 0; j < 16; ++j) f[j >> 3][j & 7] = *(const f32x4*)(src + (size_t)j * NN);
#pragma unroll
                for (int g = 0; g < 2; ++g)
#pragma unroll
                    for (int c = 0; c < 4; ++c)
                        h[g * 4 + c] = cvt8(f32x4{f[g][0][c], f[g][1][c], f[g][2][c], f[g][3][c]},
                                            f32x4{f[g][4][c], f[g][5][c], f[g][6][c], f[g][7][c]});
            } else {
                const float* src = p.x + (size_t)(b.grp * 1024 + b.lb * 128 + hb * 64 + (lane & 31)) * KS + b.t * 64 + (lane >> 5) * 32;
#pragma unroll
                for (int j = 0; j < 16; ++j) f[j >> 3][j & 7] = *(const f32x4*)(src + (size_t)(j >> 3) * 32 * KS + (j & 7) * 4);
#pragma unroll
                for (int g = 0; g < 2; ++g)
#pragma unroll
                    for (int q = 0; q < 4; ++q) h[g * 4 + q] = cvt8(f[g][2 * q], f[g][2 * q + 1]);
            }
        }
        next_call();
        const int nd = b.isw ? 4 : 2, blk = b.isw * 8 + b.lb;
        const int dl = (b.isw ? b.grp + 2 * lane : b.grp * 2 + lane) & 7;
        const size_t doff = OFF_I + (b.t * 8 + rank) * KC_B + blk * 16384 + hb * 8192 + lane * 16;
        const size_t foff = OFF_XF + blk * 512 + (b.t * 8 + rank) * 8 + hb * 4;
        if (o < 96) {
            for (int i = 0; i < nd; ++i) {
                const int ii = (i + b.lb + hb) & (nd - 1);
                char* dst = p.ws[(b.isw ? b.grp + 2 * ii : b.grp * 2 + ii) & 7] + doff;
#pragma unroll
                for (int c = 0; c < 8; ++c) st16_sys(dst + c * 1024, h[c]);
            }
            vm_wait0();
            if (lane < nd) st_sys(p.ws[dl] + foff, call);
        } else {
            if (b.grp == (b.isw ? rank & 1 : rank >> 1)) {
#pragma unroll
                for (int c = 0; c < 8; ++c) st16_loc(myws + doff + c * 1024, h[c]);
            }
            unsigned mx = 0, se = 0;
#pragma unroll
            for (int c = 0; c < 8; ++c) {
                *(u32x4*)(reg + c * 1024 + lane * 16) = h[c];
#pragma unroll
                for (int j = 0; j < 4; ++j) {
                    const unsigned e = (h[c][j] >> 10) & 0x001f001fu;
                    mx = pk_max(mx, e);
                    se += e;
                }
            }
            mx = max(mx >> 16, mx & 0xffffu);
#pragma unroll
            for (int d = 32; d > 0; d >>= 1) mx = max(mx, (unsigned)__shfl_xor((int)mx, d));
            wave_scan<11>((se >> 16) + (se & 0xffffu), se);
            const unsigned lo = max(max((int)__builtin_amdgcn_readfirstlane(mx) - 10, (int)(se >> 12) - 6), 0), base = lo * 0x04000400u;
            unsigned q[32], sb[8], sl[8], cnt = 0;
            unsigned em[2] = {0, 0};
#pragma unroll
            for (int c = 0; c < 8; ++c) {
                unsigned acc = 0, ls = 0;
#pragma unroll
                for (int j = 3; j >= 0; --j) {
                    const unsigned wd = h[c][j], ea = wd & 0x7c007c00u, zq = pk_subsat(ea, base);
                    const unsigned c2 = pk_min(pk_sub(zq, 0x18001800u), 0x0c000c00u), l = (c2 & (c2 >> 1)) >> 10;
                    const unsigned c3 = ((zq >> 10) - 3 * ((zq >> 13) & 0x00010001u)) & (l * 7), ln = l * 3;
                    q[c * 4 + j] = (wd & 0x83ff83ffu) | c2;
                    em[c >> 2] |= pk_min(pk_subsat(base, ea), 0x00010001u) << ((c & 3) * 4 + j);
                    acc = (acc << (ln >> 16)) | (c3 >> 16);
                    acc = (acc << (ln & 31)) | (c3 & 7);
                    ls += l;
                }
                sb[c] = acc;
                sl[c] = cnt;
                cnt += (ls >> 16) + (ls & 0xffffu);
            }
            unsigned nesc;
            const unsigned pre = wave_scan<7>(cnt, nesc);
            u64 m = ((u64)em[1] << 32) | em[0];
            cfence();
            unsigned ne = 0;
            bool raw = false;
            for (u64 bal; (bal = __builtin_amdgcn_ballot_w64(m != 0)) != 0 && !raw;) {
                bool far = false;
                if (m != 0) {
                    const unsigned bt = __builtin_ctzll(m), wi = (bt & 15) + (bt >> 5) * 16;
                    m &= m - 1;
                    const unsigned idx = (wi >> 2) * 512 + lane * 8 + (wi & 3) * 2 + ((bt >> 4) & 1);
                    const unsigned dlt = lo - 1 - ((*(const unsigned short*)(reg + idx * 2) >> 10) & 31u), pos = ne + mbcnt64(bal);
                    far = dlt > 15;
                    if (pos < 256) *(unsigned short*)(reg + EXC_OFF + pos * 2) = (unsigned short)(idx | (dlt << 12));
                }
                ne += __builtin_popcountll(bal);
                raw = ne > 256 || __builtin_amdgcn_ballot_w64(far) != 0;
            }
            cfence();
            char* dst = myws + OFF_P + (size_t)((b.t * 64 + (b.isw ? 32 + b.grp * 16 : b.grp * 8) + b.lb) * 2 + hb) * 8192 + lane * 16;
            unsigned ext = (16 * ne + 3 * nesc + 127) >> 7;
            if (raw || ext > 96) {
                raw = true;
                ne = ext = 0;
#pragma unroll
                for (int c = 0; c < 8; ++c) st16_sys(dst + c * 1024, h[c]);
            } else {
#pragma unroll
                for (int g = 0; g < 2; ++g) {
#pragma unroll
                    for (int i = 0; i < 12; ++i) q[g * 13 + i] |= (q[26 + g * 3 + (i >> 2)] << (12 - 3 * (i & 3))) & 0x70007000u;
#pragma unroll
                    for (int i = 0; i < 3; ++i) q[g * 13 + 12] |= (q[26 + g * 3 + i] & 0x80008000u) >> (3 - i);
                }
#pragma unroll
                for (int c = 0; c < 6; ++c) st16_sys(dst + c * 1024, u32x4{q[c * 4], q[c * 4 + 1], q[c * 4 + 2], q[c * 4 + 3]});
                u64 ex = *(const u64*)(reg + EXC_OFF + lane * 8);
                const int nx = (int)ne - lane * 4;
                ex = nx >= 4 ? ex : nx <= 0 ? 0 : ex & ((1ull << (16 * nx)) - 1);
                *(u64*)(reg + lane * 8) = ((u64)q[25] << 32) | q[24];
                *(u64*)(reg + 512 + lane * 8) = ex;
                *(u32x4*)(reg + 1024 + lane * 16) = u32x4{0, 0, 0, 0};
                cfence();
#pragma unroll
                for (int c = 0; c < 8; ++c) put_bits(reg + 512, 16 * ne + 3 * (pre + sl[c]), sb[c]);
                cfence();
                if (lane < 32 + ext) st16_sys(dst + 6144, *(const u32x4*)(reg + lane * 16));
                if (lane + 32 < ext) st16_sys(dst + 7168, *(const u32x4*)(reg + 1024 + lane * 16));
            }
            vm_wait0();
            const int item = hb + 2 * (b.lb >= 8 ? 112 + b.t * 8 + b.lb - 8
                                       : b.lb < G1 ? (b.t - 1) * (8 + G1) + b.lb * 2 + b.isw
                                       : b.isw ? (b.t - 1) * (8 + G1) + G1 + b.lb : 7 * (8 + G1) + (b.t - 1) * (8 - G1) + b.lb - G1);
            if (lane < nd) {
                if (dl == rank) st_sys(myws + foff, call);
                else st_sys64(p.ws[dl] + OFF_OF + (rank * NITEM + item) * 8, ((u64)call << 32) | ne | (ext << 9) | (lo << 16) | ((unsigned)raw << 24));
            }
        }
    }

    if (u < 16) {
        const int stage = u >> 3, ul = u & 7;
        const int rb = (xcd >> 1) * 2 + (ul >> 2);
        const int cb = stage * 8 + (xcd & 1) * 4 + (ul & 3);
        const int wm = wv & 3, kh = wv >> 2;
        const unsigned xoff = rb * 16384 + (wm * 2 + kh) * 2048 + lane * 16, woff = (8 + cb) * 16384 + wv * 2048 + lane * 16;
        const int fpos = (lane & ~7) | ((rank + lane) & 7);
        const char* const xf = myws + OFF_XF + rb * 512 + fpos * 8;
        const char* const wf = myws + OFF_XF + (8 + cb) * 512 + fpos * 8;
        u64 ready = 0;
        unsigned timed_out = 0;
        auto ensure = [&](int n) {
            if ((ready >> n) & 1) return;
            const u64 t0 = __builtin_amdgcn_s_memrealtime();
            for (unsigned spin = 1;; ++spin) {
                const u64 a = ld_sys64(xf), b = ld_sys64(wf);
                ready = __builtin_amdgcn_ballot_w64(min(min((unsigned)a, (unsigned)(a >> 32)), min((unsigned)b, (unsigned)(b >> 32))) >= call);
                if ((ready >> n) & 1) break;
                if ((spin & 255) == 0 && (__builtin_amdgcn_readfirstlane(ld_agt(tmo)) == call ||
                                          __builtin_amdgcn_s_memrealtime() - t0 > DEADLINE)) {
                    if (lane == 0) st_agt(tmo, call);
                    vm_wait0();
                    ready = ~0ull;
                    timed_out = 1;
                    break;
                }
                __builtin_amdgcn_s_sleep(8);
            }
        };
        f16x8 xa[NSLOT][2];
        auto kbase = [&](int n) { return myws + OFF_I + (size_t)((n & ~7) | ((rank + n) & 7)) * KC_B; };
        auto issue_w = [&](int n, int slot) { glds2k(kbase(n), woff, lds0 + slot * 16384 + wv * 2048); };
        auto issue_x = [&](int n, f16x8 (&d)[2]) {
            asm volatile("global_load_dwordx4 %0, %2, %3 " GSC "\n\tglobal_load_dwordx4 %1, %2, %3 offset:1024 " GSC
                         : "+v"(d[0]), "+v"(d[1]) : "v"(xoff), "s"(kbase(n)) : "memory");
        };
        struct Frag { f16x8 b[2][4]; };
        auto rd = [&](Frag& f, int slot) {
            const char* sa = lds + slot * 16384 + kh * 8192 + lane * 16;
#pragma unroll
            for (int i = 0; i < 8; ++i) f.b[i >> 2][i & 3] = *(const f16x8*)(sa + i * 1024);
        };
        f32x16 acc[4];
#pragma unroll
        for (int c = 0; c < 4; ++c)
#pragma unroll
            for (int r = 0; r < 16; ++r) acc[c][r] = 0.f;

        __builtin_amdgcn_s_waitcnt(0);
        if (wv == 0) { for (int n = 0; n < NSLOT; ++n) ensure(n); }
        __builtin_amdgcn_s_barrier();
        asm volatile("" ::: "memory");
#pragma unroll
        for (int n = 0; n < NSLOT; ++n) {
            xa[n][0] = xa[n][1] = f16x8{};
            issue_w(n, n);
            issue_x(n, xa[n]);
        }
        asm volatile("s_waitcnt vmcnt(%0)" :: "i"(4 * (NSLOT - 1)) : "memory");
        __builtin_amdgcn_s_barrier();
        asm volatile("" ::: "memory");
        Frag fr[2];
        rd(fr[0], 0);
        auto steps = [&](int n0, auto last) {
#pragma unroll
            for (int i = 0; i < NSLOT; ++i) {
                if (!last && wv == 0) ensure(n0 + i + NSLOT);
                asm volatile("s_waitcnt vmcnt(%2)" : "+v"(xa[i][0]), "+v"(xa[i][1])
                             : "i"(last ? (i < NSLOT - 2 ? 4 * (NSLOT - 2 - i) : 0) : 4 * (NSLOT - 2)) : "memory");
                __builtin_amdgcn_s_waitcnt(0xC07F);
                __builtin_amdgcn_s_barrier();
                asm volatile("" ::: "memory");
                if (!last || i + 1 < NSLOT) rd(fr[~i & 1], (i + 1) % NSLOT);
                if (!last) issue_w(n0 + i + NSLOT, i);
                __builtin_amdgcn_sched_barrier(0);
#pragma unroll
                for (int qq = 0; qq < 2; ++qq)
#pragma unroll
                    for (int c = 0; c < 4; ++c)
                        acc[c] = __builtin_amdgcn_mfma_f32_32x32x16_f16(xa[i][qq], fr[i & 1].b[qq][c], acc[c], 0, 0, 0);
                __builtin_amdgcn_sched_barrier(0);
                if (!last) issue_x(n0 + i + NSLOT, xa[i]);
            }
        };
        for (int n = 0; n < 64 - NSLOT; n += NSLOT) steps(n, std::false_type{});
        unsigned ord = 0;
        if (tid == 0) ord = __hip_atomic_fetch_add((gu32*)(myws + OFF_ORD), 1u, __ATOMIC_RELAXED, __HIP_MEMORY_SCOPE_AGENT);
        steps(64 - NSLOT, std::true_type{});
        __syncthreads();
        f32x4 v[8];
        {
            char* dst = lds + (wv ^ 4) * REG_B + lane * 16;
            if (kh == 0) {
#pragma unroll
                for (int r = 0; r < 8; ++r) {
                    *(f32x4*)(dst + r * 1024) = f32x4{acc[0][8 + r], acc[1][8 + r], acc[2][8 + r], acc[3][8 + r]};
                    v[r] = f32x4{acc[0][r], acc[1][r], acc[2][r], acc[3][r]};
                }
            } else {
#pragma unroll
                for (int r = 0; r < 8; ++r) {
                    *(f32x4*)(dst + r * 1024) = f32x4{acc[0][r], acc[1][r], acc[2][r], acc[3][r]};
                    v[r] = f32x4{acc[0][8 + r], acc[1][8 + r], acc[2][8 + r], acc[3][8 + r]};
                }
            }
        }
        if (tid == 0) *(unsigned*)(lds + LDS_BAD) = (ord & 127) | (timed_out << 7);
        __syncthreads();
        ord = __builtin_amdgcn_readfirstlane(*(const unsigned*)(lds + LDS_BAD));
        const bool bad = ord >> 7;
        ord &= 127;
#pragma unroll
        for (int r = 0; r < 8; ++r) v[r] += *(const f32x4*)(reg + r * 1024 + lane * 16);
        if (bad) {
#pragma unroll
            for (int r = 0; r < 8; ++r) v[r] = f32x4{__builtin_nanf(""), __builtin_nanf(""), __builtin_nanf(""), __builtin_nanf("")};
        }
        asm volatile("s_waitcnt lgkmcnt(0)" ::: "memory");
        u32x32 x;
        unsigned se = 0, nz = 0;
#pragma unroll
        for (int j = 31; j >= 0; --j) {
            const float val = v[j >> 2][j & 3];
            x[j] = val <= 0.f ? 0u : __float_as_uint(val);
            se += x[j] >> 23;
            nz = (nz << 1) | min(x[j], 1u);
        }
        se += __builtin_popcount(nz) << 20;
#pragma unroll
        for (int o = 32; o > 0; o >>= 1) se += (unsigned)__shfl_xor((int)se, o);
        const unsigned elo = __builtin_amdgcn_readfirstlane(
            min(max((int)((float)(se & 0xfffffu) * __builtin_amdgcn_rcpf((float)max(se >> 20, 1u))) - 6, 1), 245));
        unsigned pl[32], sb[4], sl[4], bits = 0, hb = 0, ne = 0;
#pragma unroll
        for (int g = 3; g >= 0; --g) {
            unsigned acc = 0, ls = 0;
#pragma unroll
            for (int j = g * 8 + 7; j >= g * 8; --j) {
                const unsigned z = min((x[j] >> 23) - elo, 11u), in = ubfe(0x7ffu, z, 1), esc = ubfe(0x63fu, z, 1), ln = __umul24(esc, 3);
                pl[j] = (x[j] & ~0x800000u) | ((0xfd6000u << z) & 0x800000u);
                bits = (bits << 1) | in;
                hb = (hb << in) | ubfe(0x73fu, z, 1);
                acc = (acc << ln) | ubfe(0x4003eb1au, __umul24(z, 3), ln);
                ls += esc;
            }
            sb[g] = acc;
            sl[g] = ls;
        }
        nz &= ~bits;
        for (u64 bal = __builtin_amdgcn_ballot_w64(nz != 0); bal && ne <= EXC_MAX; bal &= bal - 1) {
            const unsigned l = __builtin_ctzll(bal);
            for (unsigned m = __builtin_amdgcn_readlane(nz, l); m; m &= m - 1, ++ne) {
                const unsigned j = __builtin_ctz(m);
                if (lane == l) *(u64*)(reg + 256 + ne * 8) = ((u64)x[j] << 32) | (l * 32 + j);
            }
        }
#pragma unroll
        for (int j = 0; j < 32; ++j) v[j >> 2][j & 3] = __uint_as_float(x[j]);
        unsigned off, nesc;
        const unsigned pre = wave_scan<6>(__builtin_popcount(bits), off);
        unsigned ep = wave_scan<6>(sl[0] + sl[1] + sl[2] + sl[3], nesc);
        const unsigned a16 = 256 + ne * 8, bs = (a16 + 3 * off + 3) & ~3u;
        unsigned n16 = (bs * 8 + off + 3 * nesc + 127) >> 7;
        const int tile = stage * 64 + rb * 8 + (cb & 7), ts = wm * 2 + kh;
        char* const dst = myws + OFF_O + (size_t)(tile * 8 + ts) * SLOT_B;
        if (ne > EXC_MAX || n16 >= 512) {
            ne = n16 = 0;
#pragma unroll
            for (int r = 0; r < 8; ++r) st16_sys(dst + r * 1024 + lane * 16, __builtin_bit_cast(u32x4, v[r]));
        } else {
            *(unsigned*)(reg + lane * 4) = bits;
#pragma unroll
            for (int i = 0; i < 4; ++i) *(unsigned*)(reg + bs + i * 256 + lane * 4) = 0;
            unsigned pos = pre;
#pragma unroll
            for (int j = 0; j < 32; ++j) {
                const unsigned t = __builtin_amdgcn_sbfe(bits, j, 1), at = (pos & t) | (4200u & ~t);
                *(unsigned short*)(reg + a16 + at * 2) = (unsigned short)pl[j];
                *(unsigned char*)(reg + a16 + off * 2 + at) = (unsigned char)(pl[j] >> 16);
                pos -= t;
            }
            cfence();
            put_bits(reg + bs, pre, hb);
#pragma unroll
            for (int g = 0; g < 4; ++g) {
                put_bits(reg + bs, off + 3 * ep, sb[g]);
                ep += sl[g];
            }
            cfence();
            for (unsigned i = lane; i < n16; i += 64) st16_sys(dst + i * 16, *(const u32x4*)(reg + i * 16));
        }
        const u64 flagv = ((u64)call << 32) | n16 | (ne << 9) | (elo << 16) | ((unsigned)tile << 25);
        vm_wait0();
        if (lane < NR && lane != rank) st_sys64(p.ws[lane] + OFF_OF + (rank * NITEM + NIN + ord * 8 + ts) * 8, flagv);
        float* o = p.out + (size_t)((rank >> 1) * 1024 + rb * 128 + wm * 32 + kh * 16 + (lane >> 5) * 4) * NN
                   + (rank & 1) * 2048 + cb * 128 + (lane & 31) * 4;
#pragma unroll
        for (int r = 0; r < 8; ++r) st16_wt(o + (size_t)((r & 3) + 8 * (r >> 2)) * NN, v[r]);
        return;
    }

    {
        __builtin_amdgcn_s_waitcnt(0);
        const int s = xcd;
        if (s == rank || wv >= EW) return;
        unsigned* const tok = (unsigned*)(lds + LDS_TOK);
        bool bad = false;
        unsigned tk = 0;
        auto ticket = [&]() {
            if (lane == 0) tk = __hip_atomic_fetch_add((gu32*)(myws + OFF_ETK + xcd * 128), 1u, __ATOMIC_RELAXED, __HIP_MEMORY_SCOPE_AGENT);
        };
        auto pull = [&](const char* src, unsigned n16, unsigned wgt) {
            unsigned my = 0;
            if (lane == 0) my = __hip_atomic_fetch_add(tok, wgt, __ATOMIC_RELAXED, __HIP_MEMORY_SCOPE_WORKGROUP);
            my = __builtin_amdgcn_readfirstlane(my) + wgt;
            while ((int)(my - __builtin_amdgcn_readfirstlane(__hip_atomic_load(tok + 1, __ATOMIC_RELAXED, __HIP_MEMORY_SCOPE_WORKGROUP))) > TOK)
                __builtin_amdgcn_s_sleep(1);
            src = uni_ptr(src);
            for (unsigned it = 0; it * 64 < n16; ++it)
                if (it * 64 + lane < n16) glds1k(src + it * 1024, lane * 16, lds0 + wv * REG_B + it * 1024);
            if (n16 > 384) asm volatile("s_waitcnt vmcnt(2)" ::: "memory");
            else vm_wait0();
            if (lane == 0) __hip_atomic_fetch_add(tok + 1, wgt, __ATOMIC_RELAXED, __HIP_MEMORY_SCOPE_WORKGROUP);
            vm_wait0();
        };
        ticket();
        for (;;) {
            const unsigned k = __builtin_amdgcn_readfirstlane(tk) - (call - 1) * (NITEM + 16 * EW);
            if (k >= NITEM) break;
            const char* const fp = myws + OFF_OF + (s * NITEM + k) * 8;
            unsigned meta = ((k - NIN) >> 3) << 25;
            if (!bad) {
                const u64 t0 = __builtin_amdgcn_s_memrealtime();
                for (unsigned spin = 1;; ++spin) {
                    const u64 fl = ld_sys64(fp);
                    if (__builtin_amdgcn_readfirstlane((unsigned)(fl >> 32)) >= call) { meta = __builtin_amdgcn_readfirstlane((unsigned)fl); break; }
                    if ((spin & 255) == 0 && (__builtin_amdgcn_readfirstlane(ld_agt(tmo)) == call ||
                                              __builtin_amdgcn_s_memrealtime() - t0 > DEADLINE)) {
                        if (lane == 0) st_agt(tmo, call);
                        bad = true;
                        break;
                    }
                    __builtin_amdgcn_s_sleep(8);
                }
            }
            if (k < NIN) {
                if (bad) { ticket(); continue; }
                const int hb = k & 1, kb = k >> 1;
                int isw = 1, t = (kb - 112) >> 3, lb = 8 + (kb & 7);
                if (kb < 7 * (8 + G1)) {
                    const int e = kb % (8 + G1);
                    t = kb / (8 + G1) + 1;
                    isw = e < 2 * G1 ? e & 1 : 1;
                    lb = e < 2 * G1 ? e >> 1 : e - G1;
                } else if (kb < 112) {
                    t = (kb - 7 * (8 + G1)) / (8 - G1) + 1;
                    isw = 0;
                    lb = G1 + (kb - 7 * (8 + G1)) % (8 - G1);
                }
                const unsigned ne = min(meta & 511u, 256u), base = ((meta >> 16) & 31u) * 0x04000400u;
                const bool raw = (meta >> 24) & 1;
                const char* src = p.ws[s] + OFF_P + (size_t)((t * 64 + (isw ? 32 + (rank & 1) * 16 : (rank >> 1) * 8) + lb) * 2 + hb) * 8192;
                pull(src, raw ? 512 : 416 + min((meta >> 9) & 127u, 96u), 2);
                ticket();
                u32x4 h[8];
                if (!raw) {
                    const u64 ex = *(const u64*)(reg + 6656 + lane * 8), q2 = *(const u64*)(reg + 6144 + lane * 8);
                    unsigned q[32], sl[8], cnt = 0, tot;
#pragma unroll
                    for (int c = 0; c < 6; ++c) {
                        const u32x4 t4 = *(const u32x4*)(reg + c * 1024 + lane * 16);
#pragma unroll
                        for (int j = 0; j < 4; ++j) q[c * 4 + j] = t4[j];
                    }
                    q[24] = (unsigned)q2;
                    q[25] = (unsigned)(q2 >> 32);
#pragma unroll
                    for (int i = 0; i < 6; ++i) {
                        const unsigned* cr = q + (i / 3) * 13;
                        q[26 + i] = (cr[12] << (3 - i % 3)) & 0x80008000u;
#pragma unroll
                        for (int j = 0; j < 4; ++j) q[26 + i] |= (cr[(i % 3) * 4 + j] & 0x70007000u) >> (12 - 3 * j);
                    }
#pragma unroll
                    for (int c = 0; c < 8; ++c) {
                        sl[c] = cnt;
#pragma unroll
                        for (int j = 0; j < 4; ++j) cnt += __builtin_popcount(q[c * 4 + j] & (q[c * 4 + j] >> 1) & 0x04000400u);
                    }
                    const unsigned pre = wave_scan<7>(cnt, tot);
#pragma unroll
                    for (int c = 0; c < 8; ++c) {
                        unsigned acc = get_bits(reg + 6656, 16 * ne + 3 * (pre + sl[c]));
#pragma unroll
                        for (int j = 0; j < 4; ++j) {
                            const unsigned wd = q[c * 4 + j], c2 = (wd >> 10) & 0x00030003u, l = c2 & (c2 >> 1), ln = l * 3, mk = l * 15;
                            unsigned c3 = acc & 7;
                            acc >>= ln & 31;
                            c3 |= (acc & 7) << 16;
                            acc >>= ln >> 16;
                            c3 += 3 * ((c3 >> 1) & (c3 >> 2) & 0x00010001u);
                            h[c][j] = (wd & 0x83ff83ffu) + (((c3 & mk) | ((c2 + 0x00060006u) & ~mk)) << 10) + base;
                        }
                    }
                    cfence();
#pragma unroll
                    for (int c = 0; c < 8; ++c) *(u32x4*)(reg + c * 1024 + lane * 16) = h[c];
                    cfence();
#pragma unroll
                    for (int j = 0; j < 4; ++j) {
                        const unsigned en = (unsigned)(ex >> (16 * j)) & 0xffffu;
                        if (lane * 4 + j < ne)
                            __hip_atomic_fetch_sub((unsigned*)(reg + (en & 4094u) * 2), ((en >> 12) + 1) << (10 + 16 * (en & 1)),
                                                   __ATOMIC_RELAXED, __HIP_MEMORY_SCOPE_WAVEFRONT);
                    }
                }
                cfence();
                const int blk = isw * 8 + lb;
                char* dst = myws + OFF_I + (t * 8 + s) * KC_B + blk * 16384 + hb * 8192 + lane * 16;
#pragma unroll
                for (int c = 0; c < 8; ++c) h[c] = *(const u32x4*)(reg + c * 1024 + lane * 16);
#pragma unroll
                for (int c = 0; c < 8; ++c) st16_loc(dst + c * 1024, h[c]);
                vm_wait0();
                if (lane == 0) st_sys(myws + OFF_XF + blk * 512 + (t * 8 + s) * 8 + hb * 4, call);
                continue;
            }
            const int strip = (meta >> 25) * 8 + (k & 7);
            f32x4 v[8];
            if (!bad) {
                const unsigned n16 = meta & 511u, ne = min((meta >> 9) & 127u, (unsigned)EXC_MAX), elo = (meta >> 16) & 255u;
                pull(p.ws[s] + OFF_O + (size_t)strip * SLOT_B, n16 ? n16 : 512, 1);
                ticket();
                if (n16) {
                    const unsigned bits = *(const unsigned*)(reg + lane * 4);
                    const u64 ex = *(const u64*)(reg + 256 + lane * 8);
                    unsigned off, nesc, sl[4], es[32];
                    unsigned pos = wave_scan<6>(__builtin_popcount(bits), off);
                    const unsigned a16 = 256 + ne * 8, bs = (a16 + 3 * off + 3) & ~3u;
                    unsigned hb = get_bits(reg + bs, pos);
                    u32x32 x;
#pragma unroll
                    for (int j = 0; j < 32; ++j) {
                        x[j] = *(const unsigned short*)(reg + a16 + pos * 2) | (*(const unsigned char*)(reg + a16 + off * 2 + pos) << 16);
                        pos += (bits >> j) & 1;
                    }
#pragma unroll
                    for (int j = 0; j < 32; ++j) {
                        const unsigned in = ubfe(bits, j, 1), h = hb & in;
                        es[j] = h & (x[j] >> 23);
                        x[j] |= h << 24;
                        hb >>= in;
                        sl[j >> 3] = ((j & 7) ? sl[j >> 3] : 0u) + es[j];
                    }
                    unsigned ep = wave_scan<6>(sl[0] + sl[1] + sl[2] + sl[3], nesc);
#pragma unroll
                    for (int g = 0; g < 4; ++g) {
                        unsigned acc = get_bits(reg + bs, off + 3 * ep);
                        ep += sl[g];
#pragma unroll
                        for (int j = g * 8; j < g * 8 + 8; ++j) {
                            const unsigned ln = __umul24(es[j], 3), d = ubfe(0x56789a01u, ubfe(acc, 0, ln) << 2, 4);
                            acc >>= ln;
                            x[j] = (x[j] + ((elo + 7 - d) << 23)) & __builtin_amdgcn_sbfe(bits, j, 1);
                        }
                    }
                    for (unsigned e = 0; e < ne; ++e) {
                        const unsigned id = __builtin_amdgcn_readlane((unsigned)ex, e), val = __builtin_amdgcn_readlane((unsigned)(ex >> 32), e);
                        x[id & 31] = lane == (id >> 5) ? val : x[id & 31];
                    }
#pragma unroll
                    for (int j = 0; j < 32; ++j) v[j >> 2][j & 3] = __uint_as_float(x[j]);
                } else {
#pragma unroll
                    for (int r = 0; r < 8; ++r) v[r] = *(const f32x4*)(reg + r * 1024 + lane * 16);
                }
                cfence();
            } else {
                ticket();
#pragma unroll
                for (int r = 0; r < 8; ++r) v[r] = f32x4{__builtin_nanf(""), __builtin_nanf(""), __builtin_nanf(""), __builtin_nanf("")};
            }
            const int tile = strip >> 3, rb = (tile >> 3) & 7, cb = (tile >> 6) * 8 + (tile & 7);
            float* o = p.out + (size_t)((s >> 1) * 1024 + rb * 128 + (strip & 7) * 16 + (lane >> 5) * 4) * NN
                       + (s & 1) * 2048 + cb * 128 + (lane & 31) * 4;
#pragma unroll
            for (int r = 0; r < 8; ++r) st16_wt(o + (size_t)((r & 3) + 8 * (r >> 2)) * NN, v[r]);
        }
    }
}

extern "C" void kernel_launch(int rank, int nranks, ncclComm_t comm,
                              hipStream_t stream,
                              void* const* d_in, const int* in_sizes, int n_in,
                              void* d_out, int out_size,
                              const int* in_dtypes, int out_dtype,
                              void* const* d_out_all,
                              void* const* d_ws_all, size_t ws_bytes) {
    if (nranks != NR || (unsigned)rank >= (unsigned)NR || ws_bytes < WS_NEED || n_in < 2 ||
        in_sizes[0] != MM * KS || in_sizes[1] != KS * NN || out_size != MM * NN) {
        fprintf(stderr, "kernel_launch: unexpected problem shape\n");
        return;
    }
    HIP_CHECK(hipFuncSetAttribute((const void*)gemm_ar_kernel, hipFuncAttributeMaxDynamicSharedMemorySize, LDS_BYTES));
    Params p;
    p.x = (const float*)d_in[0];
    p.w = (const float*)d_in[1];
    p.out = (float*)d_out;
    for (int i = 0; i < NR; ++i) p.ws[i] = (char*)d_ws_all[i];
    p.rank = rank;
    hipLaunchKernelGGL(gemm_ar_kernel, dim3(NWG), dim3(NTHR), LDS_BYTES, stream, p);
    HIP_CHECK(hipGetLastError());
}
```
